```python
import numpy as np

kernel_name = "gpu_ag_gemm_m4096_k4096_n4096_e4m3e4m3_dqsilu"


ABSMAX_RTOL = 0.03

COMPUTE_DTYPE = "float32"
EPILOGUE = "dequant_silu"
IN_QUANT = None
OUT_QUANT = None
IN_FP8 = "e4m3e4m3"
IN_DTYPES = {"x": "e4m3", "w_mat": "e4m3",
             "scale_x": "float32", "scale_w": "float32"}
WORLD_SIZE = 8

def _snap_fp8(x, mant_bits, max_abs, min_normal):
    a = np.abs(x.astype(np.float64))
    _m, _e = np.frexp(a)
    q = np.where(a >= min_normal, np.ldexp(1.0, _e - 1 - mant_bits),
                 min_normal * 2.0 ** -mant_bits)
    snapped = np.minimum(np.round(a / q) * q, max_abs)
    return (np.sign(x) * snapped).astype(np.float32)

def setup_inputs():

    x = _snap_fp8(np.random.randn(4096, 4096).astype(np.float32), 3, 448.0, 2.0 ** -6)
    w_mat = _snap_fp8(np.random.randn(4096, 4096).astype(np.float32), 3, 448.0, 2.0 ** -6)
    scale_x = (0.02 * (1.0 + np.random.rand(1))).astype(np.float32)
    scale_w = (0.02 * (1.0 + np.random.rand(1))).astype(np.float32)
    return {"x": x, "w_mat": w_mat, "scale_x": scale_x, "scale_w": scale_w}

def reference(x, w_mat, scale_x, scale_w):
    acc = (x @ w_mat).astype(np.float32)
    y = (acc * (scale_x * scale_w)).astype(np.float32)
    return (y / (1.0 + np.exp(-np.clip(y, -60.0, 60.0)))).astype(np.float32)
```

```cpp
#include <hip/hip_runtime.h>
#include <rccl/rccl.h>
#include <cstdio>

#define HIP_CHECK(expr) do { hipError_t _e = (expr); if (_e != hipSuccess) { \
    fprintf(stderr, "HIP error %s at %s:%d\n", hipGetErrorString(_e), __FILE__, __LINE__); \
    return; } } while (0)

typedef unsigned u32;
typedef unsigned long long u64;
typedef u32 u32x4 __attribute__((ext_vector_type(4)));
typedef int v2i __attribute__((ext_vector_type(2)));
typedef int v4i __attribute__((ext_vector_type(4)));
typedef int v8i __attribute__((ext_vector_type(8)));
typedef float v4f __attribute__((ext_vector_type(4)));
#define LDS __attribute__((address_space(3)))
#define GLB __attribute__((address_space(1)))
#define CST __attribute__((address_space(4)))

constexpr int NR = 8;
constexpr int KD = 4096, NL = 512;
constexpr u32 FRAG = 2048;
constexpr u32 LINE = 128;
constexpr u32 OFF_CNT = 0;
constexpr u32 OFF_TMO = OFF_CNT + 256 * LINE;
constexpr u32 OFF_ELECT = OFF_TMO + LINE;
constexpr u32 OFF_CLEAN = OFF_ELECT + 64 * LINE;
constexpr u32 OFF_CPY = OFF_CLEAN + 64 * LINE;
constexpr u32 OFF_PSH = OFF_CPY + 64 * LINE;
constexpr u32 OFF_XC = 1u << 20;
constexpr u32 XC_BYTES = 1024 * FRAG;
constexpr u32 OFF_XP = OFF_XC + 2 * XC_BYTES;
constexpr u32 XP_BYTES = 128 * FRAG;
constexpr u32 WS_USED = OFF_XP + 2 * NR * XP_BYTES;
#define STAMP(i) do {} while (0)
#define STAMP_INIT() do {} while (0)
#define STAMP_FLUSH() do {} while (0)
#define L_STAMP 0
constexpr u32 L_STAGE = 16384;
constexpr u32 L_XTRA = 8 * L_STAGE;
constexpr u32 L_CTL = L_XTRA + 16384;
constexpr u32 L_BYTES = L_CTL + 64 + L_STAMP;
constexpr u32 PROW = 272;
constexpr u32 PBUF = 16 * PROW;
#define PDEPTH 2
#define GOSTEP 5
#define DUALR 2
constexpr int DUAL = DUALR;
constexpr u64 DEADLINE_TICKS = 50000000ull;

struct Params {
  char* ws[NR];
  const unsigned char* x;
  const unsigned char* w;
  const float* sx;
  const float* sw;
  float* out;
  int rank;
  int pad;
};

template <int N> __device__ __forceinline__ void wait_vm() { asm volatile("s_waitcnt vmcnt(%0)" :: "n"(N) : "memory"); }
__device__ __forceinline__ void wait_lgkm() { asm volatile("s_waitcnt lgkmcnt(0)" ::: "memory"); }
__device__ __forceinline__ u32 ld_sys(const char* a) { return __hip_atomic_load((GLB u32*)a, __ATOMIC_RELAXED, __HIP_MEMORY_SCOPE_SYSTEM); }
__device__ __forceinline__ u32 ld_agent(const char* a) { return __hip_atomic_load((GLB u32*)a, __ATOMIC_RELAXED, __HIP_MEMORY_SCOPE_AGENT); }
__device__ __forceinline__ void st_agent(char* a, u32 v) { __hip_atomic_store((GLB u32*)a, v, __ATOMIC_RELAXED, __HIP_MEMORY_SCOPE_AGENT); }
__device__ __forceinline__ u32 lds_ld(LDS char* a) { return __hip_atomic_load((LDS u32*)a, __ATOMIC_RELAXED, __HIP_MEMORY_SCOPE_WORKGROUP); }

__device__ __forceinline__ void dma_w(u32 lds, u32 ve, u32 vo, u32x4 rs, u32 so) {
  u32 keep;
#define DMA_STEP(V) "s_add_u32 m0, m0, 0x400\n\ts_add_u32 %1, %1, 0x2000\n\tbuffer_load_dwordx4 " V ", %5, %1 offen lds\n\t"
  asm volatile("s_mov_b32 %0, m0\n\ts_mov_b32 m0, %2\n\ts_nop 0\n\tbuffer_load_dwordx4 %3, %5, %1 offen lds\n\t"
               DMA_STEP("%4") DMA_STEP("%3") DMA_STEP("%4") DMA_STEP("%3") DMA_STEP("%4") DMA_STEP("%3") DMA_STEP("%4")
               "s_mov_b32 m0, %0"
               : "=&s"(keep), "+s"(so) : "s"(lds), "v"(ve), "v"(vo), "s"(rs) : "memory", "scc");
#undef DMA_STEP
}
__device__ __forceinline__ void st16_sys(char* base, u32 off, u32x4 v) {
  asm volatile("global_store_dwordx4 %0, %1, %2 sc0 sc1\n\ts_nop 1" :: "v"(off), "v"(v), "s"(base) : "memory");
}
__device__ __forceinline__ bool poll_all(const char* word, u32 target, char* tmo, u32 call) {
  const u64 t0 = __builtin_amdgcn_s_memrealtime();
  while (!__all(ld_sys(word) >= target)) {
    if (ld_agent(tmo) == call || __builtin_amdgcn_s_memrealtime() - t0 > DEADLINE_TICKS) { st_agent(tmo, call); return false; }
    __builtin_amdgcn_s_sleep(2);
  }
  return true;
}

__global__ void __launch_bounds__(512, 2) ag_gemm(Params p) {
  __shared__ __attribute__((aligned(16))) char smem_[L_BYTES];
  LDS char* const smem = (LDS char*)smem_;
  const int lane = threadIdx.x & 63;
  const int w = __builtin_amdgcn_readfirstlane(threadIdx.x >> 6);
  const int b = blockIdx.x, slot = b & 7, q = b >> 3, grp = q >> 3, panel = q & 7;
  const int gq = lane >> 4, r16 = lane & 15;
  STAMP_INIT();
  STAMP(0);
  asm volatile("" :: "s"(p.ws[0]), "s"(p.ws[1]), "s"(p.ws[2]), "s"(p.ws[3]), "s"(p.ws[4]), "s"(p.ws[5]), "s"(p.ws[6]), "s"(p.ws[7]),
               "s"(p.x), "s"(p.w), "s"(p.sx), "s"(p.sw), "s"(p.out), "s"(p.rank));
  char* const me = p.ws[0];
  const char* const cntp = me + OFF_CNT + b * LINE;
  u32 call;

  const bool pushcu = (q & 15) == 0;
  const bool copycu = !pushcu && (panel < 2 || q == 2);
  const int pc = slot * 2 + (q >> 4);
  const int cc = slot * 7 + (panel == 0 ? grp >> 1 : panel == 1 ? 2 + grp : 6);
  const u64 wa = (u64)p.w;
  const u32x4 wrs = {(u32)wa, (u32)(wa >> 32) & 0xffffu, (u32)(KD * NL), 0x00020000u};
  const u32 wve = (u32)(lane >> 2) * NL + panel * 64 + (((lane & 3) ^ (gq & 1)) << 4), wvo = wve ^ 32u;
  const u32 lbase = (u32)(size_t)smem;
  const u32 s01 = w * L_STAGE;
  auto frag_src = [&](int F) {
    return p.x + (size_t)((F >> 7) * 64 + ((F >> 3) & 3) * 16 + r16) * KD + ((((F >> 5) & 3) * 8 + (F & 7)) * 128 + gq * 16);
  };
  if (threadIdx.x < 8) *(LDS u32*)(smem + L_CTL + threadIdx.x * 4) = 0u;
  STAMP(22);
  if (pushcu) {
    u32x4 x0, x1;
    asm volatile("global_load_dwordx4 %0, %3, off\n\tglobal_load_dwordx4 %1, %3, off offset:64\n\ts_load_dword %2, %4, 0x0\n\ts_waitcnt vmcnt(0) lgkmcnt(0)"
                 : "=&v"(x0), "=&v"(x1), "=&s"(call) : "v"(frag_src(w * 16 + ((pc + 2 * w) & 15))), "s"(cntp) : "memory");
    call += 1u;
    STAMP(14);
    *(LDS u32x4*)(smem + L_XTRA + w * FRAG + lane * 16) = x0;
    *(LDS u32x4*)(smem + L_XTRA + w * FRAG + 1024 + lane * 16) = x1;
    wait_lgkm();
    __builtin_amdgcn_s_barrier();
    asm volatile("" ::: "memory");
    char* const dst = p.ws[(w + 1) & 7];
    const u32 off = OFF_XP + ((call & 1u) * NR + p.rank) * XP_BYTES + lane * 16;
#pragma unroll
    for (int i = 0; i < 8; ++i) {
      const int fi = (i + w) & 7;
      const u32x4 a = *(LDS u32x4*)(smem + L_XTRA + fi * FRAG + lane * 16);
      const u32x4 c = *(LDS u32x4*)(smem + L_XTRA + fi * FRAG + 1024 + lane * 16);
      const u32 o = off + (u32)(fi * 16 + ((pc + 2 * fi) & 15)) * FRAG;
      if (i >= PDEPTH) { if (i > GOSTEP && i <= GOSTEP + PDEPTH - 1) wait_vm<2 * PDEPTH - 1>(); else wait_vm<2 * PDEPTH - 2>(); }
      if (i == GOSTEP && lane == 0) asm volatile("global_atomic_add %0, %1, off sc1" :: "v"(dst + OFF_PSH + (p.rank * 8 + 1) * LINE), "v"(1u) : "memory");
      st16_sys(dst, o, a); st16_sys(dst, o + 1024, c);
    }
    wait_vm<0>();
    STAMP(19);
    if (lane == 0) asm volatile("global_atomic_add %0, %1, off sc1" :: "v"(dst + OFF_PSH + p.rank * 8 * LINE), "v"(1u) : "memory");
  } else if (copycu) {
    u32x4 x0, x1, x2, x3;
    const int F = 128 + cc * 16 + w * 2;
    asm volatile("global_load_dwordx4 %0, %5, off\n\tglobal_load_dwordx4 %1, %5, off offset:64\n\t"
                 "global_load_dwordx4 %2, %5, off offset:128\n\tglobal_load_dwordx4 %3, %5, off offset:192\n\t"
                 "s_load_dword %4, %6, 0x0\n\ts_waitcnt vmcnt(0) lgkmcnt(0)"
                 : "=&v"(x0), "=&v"(x1), "=&v"(x2), "=&v"(x3), "=&s"(call) : "v"(frag_src(F)), "s"(cntp) : "memory");
    call += 1u;
    STAMP(14);
    const u32 off = OFF_XC + (call & 1u) * XC_BYTES + F * FRAG + lane * 16;
    st16_sys(me, off, x0); st16_sys(me, off + 1024, x1); st16_sys(me, off + 2048, x2); st16_sys(me, off + 3072, x3);
    char* wsl = p.ws[0];
#pragma unroll
    for (int i = 1; i < NR; ++i) wsl = (lane & 7) == i ? p.ws[i] : wsl;
    wsl += OFF_CPY + (p.rank * 8 + slot) * LINE;
    asm volatile("s_waitcnt vmcnt(0)" : "+v"(wsl) :: "memory");
    __builtin_amdgcn_s_barrier();
    STAMP(19);
    if (w == 0 && lane < NR) asm volatile("global_atomic_add %0, %1, off sc1" :: "v"(wsl), "v"(1u) : "memory");
  }
  dma_w(lbase + s01, wve, wvo, wrs, (u32)w * 65536u);
  dma_w(lbase + s01 + 8192, wve, wvo, wrs, (u32)(w + 8) * 65536u);
  if (!pushcu && !copycu) {
    asm volatile("s_load_dword %0, %1, 0x0\n\ts_waitcnt lgkmcnt(0)" : "=&s"(call) : "s"(cntp) : "memory");
    call += 1u;
    __builtin_amdgcn_s_barrier();
  }
  const u32 par = call & 1u;
  STAMP(1);

  v8i Wf[4][4];
  const int q8 = r16 >> 1, f2 = ((q8 >> 2) & 1) | ((gq & 1) << 1);
  const u32 tra = (u32)(gq * 16 + q8) * 64 + (r16 & 1) * 8;
  auto read_stage = [&](int j, u32 off) {
#pragma unroll
    for (int c = 0; c < 4; ++c) {
      LDS char* a = smem + off + tra + ((c ^ f2) << 4);
      v2i t0 = __builtin_amdgcn_ds_read_tr8_b64_v2i32((LDS v2i*)(a));
      v2i t1 = __builtin_amdgcn_ds_read_tr8_b64_v2i32((LDS v2i*)(a + 512));
      v2i t2 = __builtin_amdgcn_ds_read_tr8_b64_v2i32((LDS v2i*)(a + 4096));
      v2i t3 = __builtin_amdgcn_ds_read_tr8_b64_v2i32((LDS v2i*)(a + 4608));
      Wf[j][c] = v8i{t0.x, t0.y, t1.x, t1.y, t2.x, t2.y, t3.x, t3.y};
    }
  };
  wait_vm<8>();
  STAMP(15);
  read_stage(0, s01);
  wait_lgkm();
  dma_w(lbase + s01, wve, wvo, wrs, (u32)(w + 16) * 65536u);
  wait_vm<8>();
  STAMP(16);
  read_stage(1, s01 + 8192);
  wait_lgkm();
  dma_w(lbase + s01 + 8192, wve, wvo, wrs, (u32)(w + 24) * 65536u);
  wait_vm<8>();
  STAMP(17);
  read_stage(2, s01);
  wait_vm<0>();
  STAMP(18);
  read_stage(3, s01 + 8192);
  STAMP(2);
  const int sh = (p.rank + slot) & 7;
  char* const tmo = me + OFF_TMO;
  if (w == 1) {
    u32 xcc;
    asm volatile("s_getreg_b32 %0, hwreg(HW_REG_XCC_ID)" : "=s"(xcc));
    const u32 eo = ((xcc & 7u) * 8 + slot) * LINE;
    u32 cl = ld_agent(me + OFF_CLEAN + eo);
    if (cl < call) {
      u32 old = 0;
      if (lane == 0) old = __hip_atomic_fetch_max((GLB u32*)(me + OFF_ELECT + eo), call, __ATOMIC_RELAXED, __HIP_MEMORY_SCOPE_AGENT);
      old = __builtin_amdgcn_readfirstlane(old);
      if (old < call) {
        bool ok = poll_all(me + OFF_CPY + (sh * 8 + (lane & 7)) * LINE, 7u * call, tmo, call);
        STAMP(20);
        __builtin_amdgcn_fence(__ATOMIC_ACQUIRE, "");
        wait_vm<0>();
        STAMP(23);
        ok = ok && poll_all(me + OFF_PSH + (sh * 8 + 1) * LINE, 16u * call, tmo, call);
        cl = ok ? call : 0xffffffffu;
        if (lane == 0) st_agent(me + OFF_CLEAN + eo, cl);
        STAMP(21);
      } else {
        cl = poll_all(me + OFF_CLEAN + eo, call, tmo, call) ? ld_agent(me + OFF_CLEAN + eo) : 0xffffffffu;
      }
    }
    if (lane == 0) {
      if (cl != call) *(LDS u32*)(smem + L_CTL + 4) = 1u;
      __hip_atomic_store((LDS u32*)(smem + L_CTL + 20), 1u, __ATOMIC_RELAXED, __HIP_MEMORY_SCOPE_WORKGROUP);
      st_agent(me + OFF_CNT + b * LINE, call);
    }
  } else {
    while (lds_ld(smem + L_CTL + 20) == 0u) __builtin_amdgcn_s_sleep(1);
  }
  asm volatile("" ::: "memory");
  STAMP(3);

  const auto rrs = __builtin_amdgcn_make_buffer_rsrc(p.ws[slot] + OFF_XC + par * XC_BYTES, 0, XC_BYTES, 0x00020000);
  const auto lrs = __builtin_amdgcn_make_buffer_rsrc(me + OFF_XP + (par * NR + sh) * XP_BYTES, 0, XP_BYTES, 0x00020000);
  const auto ors = __builtin_amdgcn_make_buffer_rsrc(p.out, 0, 4096 * NL * 4, 0x00020000);
  const u32 fo = (u32)(grp * 8 + w) * FRAG, vo = lane * 16;
  const float scale = *(const CST float*)p.sx * *(const CST float*)p.sw;
  auto ld_r = [&](int step) {
    v4i a = __builtin_amdgcn_raw_buffer_load_b128(rrs, vo, fo + (u32)step * (32 * FRAG), 0);
    v4i c = __builtin_amdgcn_raw_buffer_load_b128(rrs, vo + 1024, fo + (u32)step * (32 * FRAG), 0);
    return v8i{a.x, a.y, a.z, a.w, c.x, c.y, c.z, c.w};
  };
  auto ld_l = [&](int j) {
    v4i a = __builtin_amdgcn_raw_buffer_load_b128(lrs, vo, fo + (u32)j * (32 * FRAG), 17);
    v4i c = __builtin_amdgcn_raw_buffer_load_b128(lrs, vo + 1024, fo + (u32)j * (32 * FRAG), 17);
    return v8i{a.x, a.y, a.z, a.w, c.x, c.y, c.z, c.w};
  };
  auto put_partial = [&](const v4f* acc, int buf) {
#pragma unroll
    for (int c = 0; c < 4; ++c) *(LDS v4f*)(smem + s01 + buf * PBUF + r16 * PROW + c * 64 + gq * 16) = acc[c];
    wait_lgkm();
    if (lane == 0) __hip_atomic_fetch_add((LDS u32*)(smem + L_CTL + 8 + buf * 4), 1u, __ATOMIC_RELAXED, __HIP_MEMORY_SCOPE_WORKGROUP);
  };
  auto reduce = [&](int r, int buf, u32 target) {
    while (lds_ld(smem + L_CTL + 8 + buf * 4) < target) __builtin_amdgcn_s_sleep(1);
    asm volatile("" ::: "memory");
    if (lane < 32) {
      const int m = 2 * w + gq;
      LDS char* a = smem + buf * PBUF + m * PROW + r16 * 16;
      v4f s = *(LDS v4f*)a;
#pragma unroll
      for (int i = 1; i < 8; ++i) s += *(LDS v4f*)(a + i * L_STAGE);
      const bool bad = lds_ld(smem + L_CTL + 4) != 0u;
      v4f o;
#pragma unroll
      for (int i = 0; i < 4; ++i) {
        const float y = s[i] * scale;
        const float z = fminf(fmaxf(y, -60.f), 60.f);
        o[i] = bad ? __builtin_nanf("") : y * __builtin_amdgcn_rcpf(1.f + __expf(-z));
      }
      const u32 off = ((u32)(sh * 512 + r * 64 + grp * 16 + m) * NL + panel * 64 + r16 * 4) * 4u;
      __builtin_amdgcn_raw_buffer_store_b128(__builtin_bit_cast(u32x4, o), ors, off, 0, 17);
    }
  };

  v8i R = ld_r(4);
  u32 pcnt = 0;
  for (int r = 1; r < 8; ++r) {
    v4f acc[4] = {}, acc0[4] = {};
#pragma unroll
    for (int j = 0; j < 4; ++j) {
      v8i cur = R;
      asm volatile("" : "+v"(cur));
      v8i L;
      if (r == DUAL) {
        if (j == 0 && !__all(pcnt >= 16u * call)) {
          if (!poll_all(me + OFF_PSH + sh * 8 * LINE, 16u * call, tmo, call) && lane == 0) *(LDS u32*)(smem + L_CTL + 4) = 1u;
        }
        asm volatile("" ::: "memory");
        L = ld_l(j);
      }
      if (r * 4 + j < 31) R = ld_r(r * 4 + j + 1);
      if (r == DUAL - 1 && j == 3) pcnt = ld_sys(me + OFF_PSH + sh * 8 * LINE);
      asm volatile("" ::: "memory");
      __builtin_amdgcn_sched_barrier(0);
      if (j == 0 && r > 1) reduce(r - 1, (r - 1) & 1, 8u * (u32)(r >> 1));
      if (j == 1 && r == DUAL + 1) reduce(0, 2, 8u);
#pragma unroll
      for (int c = 0; c < 4; ++c) acc[c] = __builtin_amdgcn_mfma_scale_f32_16x16x128_f8f6f4(Wf[j][c], cur, acc[c], 0, 0, 0, 0, 0, 0);
      if (r == DUAL) {
#pragma unroll
        for (int c = 0; c < 4; ++c) acc0[c] = __builtin_amdgcn_mfma_scale_f32_16x16x128_f8f6f4(Wf[j][c], L, acc0[c], 0, 0, 0, 0, 0, 0);
      }
    }
    put_partial(acc, r & 1);
    if (r == DUAL) put_partial(acc0, 2);
    STAMP(4 + r);
  }
  reduce(7, 1, 32u);
  STAMP(12);
  STAMP(13);
  STAMP_FLUSH();
}

extern "C" void kernel_launch(int rank, int nranks, ncclComm_t comm,
                              hipStream_t stream,
                              void* const* d_in, const int* in_sizes, int n_in,
                              void* d_out, int out_size,
                              const int* in_dtypes, int out_dtype,
                              void* const* d_out_all,
                              void* const* d_ws_all, size_t ws_bytes) {
  if (nranks != NR || (unsigned)rank >= (unsigned)NR || n_in != 4 || ws_bytes < (size_t)WS_USED + (1u << 20) ||
      in_sizes[0] != 512 * KD || in_sizes[1] != KD * NL || out_size != 4096 * NL) {
    fprintf(stderr, "unexpected problem shape\n");
    return;
  }
  Params p;
  for (int i = 0; i < NR; ++i) p.ws[i] = (char*)d_ws_all[(rank + i) & 7];
  p.x = (const unsigned char*)d_in[0];
  p.w = (const unsigned char*)d_in[1];
  p.sx = (const float*)d_in[2];
  p.sw = (const float*)d_in[3];
  p.out = (float*)d_out;
  p.rank = rank;
  p.pad = 0;
  ag_gemm<<<256, 512, 0, stream>>>(p);
  HIP_CHECK(hipGetLastError());
}
```

```cpp
#include <hip/hip_runtime.h>
#include <rccl/rccl.h>
#include <cstdio>
#include <type_traits>

#define HIP_CHECK(expr) do { hipError_t _e = (expr); if (_e != hipSuccess) { \
    fprintf(stderr, "HIP error %s at %s:%d\n", hipGetErrorString(_e), __FILE__, __LINE__); \
    return; } } while (0)

typedef unsigned u32;
typedef unsigned long long u64;
typedef u32 u32x4 __attribute__((ext_vector_type(4)));
typedef u32 u32x2 __attribute__((ext_vector_type(2)));
typedef int v2i __attribute__((ext_vector_type(2)));
typedef int v4i __attribute__((ext_vector_type(4)));
typedef int v8i __attribute__((ext_vector_type(8)));
typedef float v4f __attribute__((ext_vector_type(4)));
typedef float v2f __attribute__((ext_vector_type(2)));
#define LDS __attribute__((address_space(3)))
#define GLB __attribute__((address_space(1)))
#define CST __attribute__((address_space(4)))

constexpr int NR = 8;
constexpr int KD = 4096, NL = 512;
constexpr u32 FRAG = 2048;
constexpr u32 LINE = 128;
constexpr u32 OFF_CNT = 0;
constexpr u32 OFF_TMO = OFF_CNT + 256 * LINE;
constexpr u32 OFF_ELECT = OFF_TMO + LINE;
constexpr u32 OFF_CLEAN = OFF_ELECT + 64 * LINE;
constexpr u32 OFF_CPY = OFF_CLEAN + 64 * LINE;
constexpr u32 OFF_GO = OFF_CPY + NR * 512;
constexpr u32 OFF_PSH = OFF_GO + NR * LINE;
constexpr u32 L2_DW = 56;
constexpr u32 PK = 1024 + 512 + L2_DW * 4;
constexpr u32 STEP = 32 * PK;
constexpr u32 OFF_XC = 1u << 20;
constexpr u32 XC_RAW = 2u << 20, XC_BYTES = XC_RAW + 1024 * FRAG;
constexpr u32 OFF_XP = OFF_XC + 2 * XC_BYTES;
constexpr u32 XP_BYTES = 128 * FRAG;
constexpr u32 WS_USED = OFF_XP + 2 * NR * XP_BYTES;
static_assert(32 * STEP <= XC_RAW && STEP % 128 == 0, "layout");
#define STAMP(i) do {} while (0)
#define STAMP_INIT() do {} while (0)
#define STAMP_FLUSH() do {} while (0)
#define L_STAMP 0
constexpr u32 L_LUT = 0;
constexpr u32 L_STG = 2048, L_STAGE = 16384;
constexpr u32 L_XTRA = L_STG + 8 * L_STAGE;
constexpr u32 L_CTL = L_XTRA + 8 * FRAG;
constexpr u32 L_BYTES = L_CTL + 64 + L_STAMP;
constexpr u32 PROW = 272;
constexpr u32 PBUF = 16 * PROW;
#define PW 3
#define GOS 10
#define TOUCH 4
#define LASTAUX 17
#define OUTAUX 17
#define SR 6
#define EARLYW 0
#define DEPTH 1
#define RPOS 0
#define PRIO 0
constexpr u64 DEADLINE_TICKS = 50000000ull;

struct Params {
  char* ws[NR];
  const unsigned char* x;
  const unsigned char* w;
  const float* sx;
  const float* sw;
  float* out;
  int rank;
  int pad;
};

template <int N> __device__ __forceinline__ void wait_vm() { asm volatile("s_waitcnt vmcnt(%0)" :: "n"(N) : "memory"); }
__device__ __forceinline__ void wait_lgkm() { asm volatile("s_waitcnt lgkmcnt(0)" ::: "memory"); }
__device__ __forceinline__ u32 ld_sys(const char* a) { return __hip_atomic_load((GLB u32*)a, __ATOMIC_RELAXED, __HIP_MEMORY_SCOPE_SYSTEM); }
__device__ __forceinline__ u32 ld_agent(const char* a) { return __hip_atomic_load((GLB u32*)a, __ATOMIC_RELAXED, __HIP_MEMORY_SCOPE_AGENT); }
__device__ __forceinline__ void st_agent(char* a, u32 v) { __hip_atomic_store((GLB u32*)a, v, __ATOMIC_RELAXED, __HIP_MEMORY_SCOPE_AGENT); }
__device__ __forceinline__ u32 lds_ld(LDS char* a) { return __hip_atomic_load((LDS u32*)a, __ATOMIC_RELAXED, __HIP_MEMORY_SCOPE_WORKGROUP); }

#define WFLAGS ""
__device__ __forceinline__ void dma_w(u32 lds, u32 ve, u32 vo, u32x4 rs, u32 so) {
  u32 keep;
#define DMA_STEP(V) "s_add_u32 m0, m0, 0x400\n\ts_add_u32 %1, %1, 0x2000\n\tbuffer_load_dwordx4 " V ", %5, %1 offen" WFLAGS " lds\n\t"
  asm volatile("s_mov_b32 %0, m0\n\ts_mov_b32 m0, %2\n\ts_nop 0\n\tbuffer_load_dwordx4 %3, %5, %1 offen" WFLAGS " lds\n\t"
               DMA_STEP("%4") DMA_STEP("%3") DMA_STEP("%4") DMA_STEP("%3") DMA_STEP("%4") DMA_STEP("%3") DMA_STEP("%4")
               "s_mov_b32 m0, %0"
               : "=&s"(keep), "+s"(so) : "s"(lds), "v"(ve), "v"(vo), "s"(rs) : "memory", "scc");
#undef DMA_STEP
}
__device__ __forceinline__ void st16_sys(char* base, u32 off, u32x4 v) {
  asm volatile("global_store_dwordx4 %0, %1, %2 sc0 sc1\n\ts_nop 1" :: "v"(off), "v"(v), "s"(base) : "memory");
}
__device__ __forceinline__ void st8_sys(char* base, u32 off, u32x2 v) {
  asm volatile("global_store_dwordx2 %0, %1, %2 sc0 sc1\n\ts_nop 1" :: "v"(off), "v"(v), "s"(base) : "memory");
}
__device__ __forceinline__ void st4_sys(char* base, u32 off, u32 v) {
  asm volatile("global_store_dword %0, %1, %2 sc0 sc1\n\ts_nop 1" :: "v"(off), "v"(v), "s"(base) : "memory");
}
__device__ __forceinline__ u32 poll_all(const char* word, const char* word2, u32 target, char* tmo, u32 call) {
  const u64 t0 = __builtin_amdgcn_s_memrealtime();
  for (u32 it = 1;; ++it) {
    const u32 v = ld_sys(word), v2 = word2 ? ld_sys(word2) : v;
    if (__all(v >= target && v2 >= target)) return __builtin_amdgcn_readfirstlane(v);
    if ((it & 15u) == 0u && (ld_agent(tmo) == call || __builtin_amdgcn_s_memrealtime() - t0 > DEADLINE_TICKS)) { st_agent(tmo, call); return ~0u; }
    __builtin_amdgcn_s_sleep(1);
  }
}

constexpr u32 L2_CAP = (L2_DW * 32 - 8) / 3;
constexpr u32 HDR_RAW = 0x80;

__device__ __forceinline__ u32 wave_scan(u32 v) {
  v += __builtin_amdgcn_update_dpp(0u, v, 0x111, 0xf, 0xf, false);
  v += __builtin_amdgcn_update_dpp(0u, v, 0x112, 0xf, 0xf, false);
  v += __builtin_amdgcn_update_dpp(0u, v, 0x114, 0xf, 0xf, false);
  v += __builtin_amdgcn_update_dpp(0u, v, 0x118, 0xf, 0xf, false);
  v += __builtin_amdgcn_update_dpp(0u, v, 0x142, 0xa, 0xf, false);
  v += __builtin_amdgcn_update_dpp(0u, v, 0x143, 0xc, 0xf, false);
  return v;
}
__device__ __forceinline__ void lut_entry(u32 cb, u32& dsel, u32& csel, u32& s3) {
  u32 n = 0; dsel = 0; csel = 0x0c0c0c0cu;
#pragma unroll
  for (int b = 0; b < 4; ++b) {
    const u32 c = (cb >> (2 * b)) & 3u;
    const bool e = c == 3u;
    dsel |= (e ? 4u + n : c) << (8 * b);
    csel = e ? (csel & ~(0xffu << (8 * n))) | ((u32)b << (8 * n)) : csel;
    n += e;
  }
  s3 = 3u * n;
}
struct Packed { u32x4 n; u32x2 c; u32 l; };

__device__ __forceinline__ bool encode(const v8i& xi, Packed& o, LDS char* lutc, LDS char* buf, int lane) {
  u32 x[8];
#pragma unroll
  for (int i = 0; i < 8; ++i) x[i] = (u32)xi[i];
  u32 se = 0;
#pragma unroll
  for (int i = 0; i < 8; ++i) se = __builtin_amdgcn_sad_u8(x[i] & 0x78787878u, 0u, se);
  const u32 tot = __builtin_amdgcn_readlane(wave_scan(se), 63);
  const u32 e0 = min((tot < 512u ? 0u : tot - 512u) >> 14, 10u);
  const u32 bias = (13u - e0) * 0x01010101u;
  u32 inv = 0, cb[8], kb[8];
#pragma unroll
  for (int i = 0; i < 8; ++i) {
    const u32 gb = ((x[i] >> 3) & 0x0f0f0f0fu) + bias;
    const u32 sel = gb & 0x07070707u;
    const u32 a = __builtin_amdgcn_perm(0x02010013u, 0x0f0b0703u, sel);
    const u32 b = __builtin_amdgcn_perm(0x83838383u, 0x831f1b17u, sel);
    const u32 m = __builtin_amdgcn_perm(0u, 0u, ((gb >> 4) & 0x01010101u) | 0x0c0c0c0cu);
    const u32 code = (b & m) | (a & ~m);
    inv |= (code & 0x80808080u) | (~(gb ^ (gb >> 1)) & 0x08080808u);
    cb[i] = __builtin_amdgcn_udot4(code & 0x03030303u, 0x40100401u, 0u, false);
    kb[i] = (code >> 2) & 0x07070707u;
  }
  u64 acc = 0; u32 n3 = 0;
#pragma unroll
  for (int i = 7; i >= 0; --i) {
    const u32x2 t = *(LDS u32x2*)(lutc + cb[i] * 8u);
    const u32 comp = __builtin_amdgcn_perm(kb[i], kb[i], t.x);
    const u32 bits = __builtin_amdgcn_udot4(comp, 0x00400801u, (comp >> 15) & 0xe00u, false);
    acc = (acc << t.y) | bits;
    n3 += t.y;
  }
  const u32 incl = wave_scan(n3);
  const u32 all3 = __builtin_amdgcn_readlane(incl, 63);
  const bool fit = !__any(inv != 0u || n3 > 63u) && all3 <= 3u * L2_CAP;
  o.c = u32x2{cb[0] | (cb[1] << 8) | (cb[2] << 16) | (cb[3] << 24), cb[4] | (cb[5] << 8) | (cb[6] << 16) | (cb[7] << 24)};
#pragma unroll
  for (int j = 0; j < 4; ++j) {
    const u32 od = x[2 * j + 1] & 0x87878787u;
    o.n[j] = (x[2 * j] & 0x87878787u) | __builtin_amdgcn_alignbit(od, od, 28);
  }
  if (!fit) { o.l = HDR_RAW; return false; }
  const u32 off = incl - n3 + 8u, sh = off & 31u;
  const u64 lo = acc << sh;
  const u32 hi = ((u32)(acc >> 32) >> 1) >> (31u - sh);
  LDS u32* d = (LDS u32*)(buf + ((off >> 5) << 2));
  __hip_atomic_fetch_or(d, (u32)lo | (lane == 0 ? e0 : 0u), __ATOMIC_RELAXED, __HIP_MEMORY_SCOPE_WAVEFRONT);
  __hip_atomic_fetch_or(d + 1, (u32)(lo >> 32), __ATOMIC_RELAXED, __HIP_MEMORY_SCOPE_WAVEFRONT);
  __hip_atomic_fetch_or(d + 2, hi, __ATOMIC_RELAXED, __HIP_MEMORY_SCOPE_WAVEFRONT);
  o.l = __hip_atomic_load((LDS u32*)(buf + lane * 4), __ATOMIC_RELAXED, __HIP_MEMORY_SCOPE_WAVEFRONT);
  return true;
}

template <int B> __device__ __forceinline__ u32 byte_x8(u32 c, u32 three) {
  u32 r;
  if (B == 0) asm("v_lshlrev_b32_sdwa %0, %1, %2 dst_sel:DWORD dst_unused:UNUSED_PAD src0_sel:DWORD src1_sel:BYTE_0" : "=v"(r) : "v"(three), "v"(c));
  if (B == 1) asm("v_lshlrev_b32_sdwa %0, %1, %2 dst_sel:DWORD dst_unused:UNUSED_PAD src0_sel:DWORD src1_sel:BYTE_1" : "=v"(r) : "v"(three), "v"(c));
  if (B == 2) asm("v_lshlrev_b32_sdwa %0, %1, %2 dst_sel:DWORD dst_unused:UNUSED_PAD src0_sel:DWORD src1_sel:BYTE_2" : "=v"(r) : "v"(three), "v"(c));
  if (B == 3) asm("v_lshlrev_b32_sdwa %0, %1, %2 dst_sel:DWORD dst_unused:UNUSED_PAD src0_sel:DWORD src1_sel:BYTE_3" : "=v"(r) : "v"(three), "v"(c));
  return r;
}
__device__ __forceinline__ v8i decode(const Packed& f, u32 hdr, LDS char* lutd) {
  u32x2 t[8];
  t[0] = *(LDS u32x2*)(lutd + byte_x8<0>(f.c.x, 3u)); t[1] = *(LDS u32x2*)(lutd + byte_x8<1>(f.c.x, 3u));
  t[2] = *(LDS u32x2*)(lutd + byte_x8<2>(f.c.x, 3u)); t[3] = *(LDS u32x2*)(lutd + byte_x8<3>(f.c.x, 3u));
  t[4] = *(LDS u32x2*)(lutd + byte_x8<0>(f.c.y, 3u)); t[5] = *(LDS u32x2*)(lutd + byte_x8<1>(f.c.y, 3u));
  t[6] = *(LDS u32x2*)(lutd + byte_x8<2>(f.c.y, 3u)); t[7] = *(LDS u32x2*)(lutd + byte_x8<3>(f.c.y, 3u));
  __builtin_amdgcn_sched_barrier(0);
  const u32 n = __builtin_popcount(__builtin_amdgcn_bitop3_b32(f.c.x, f.c.x >> 1, 0x55555555u, 0x80)) +
                __builtin_popcount(__builtin_amdgcn_bitop3_b32(f.c.y, f.c.y >> 1, 0x55555555u, 0x80));
  u32 incl = wave_scan(n);
  asm("" : "+v"(incl));
  const u32 off = __umul24(incl - n, 3u) + 8u;
  const int a = (int)(off >> 3);
  const u32 w0 = __builtin_amdgcn_ds_bpermute(a, f.l), w1 = __builtin_amdgcn_ds_bpermute(a + 4, f.l), w2 = __builtin_amdgcn_ds_bpermute(a + 8, f.l);
  u64 win = ((u64)__builtin_amdgcn_alignbit(w2, w1, off) << 32) | __builtin_amdgcn_alignbit(w1, w0, off);
  const u32 e8 = (hdr & 15u) * 0x08080808u;
  const u32 td = 0x00100800u + e8, tl = (0x70686058u + e8) & 0x7f7f7f7fu, th = (0xa8a09878u + e8) & 0x7f7f7f7fu;
  v8i o;
#pragma unroll
  for (int i = 0; i < 8; ++i) {
    const u32 a0 = (u32)win;
    const u32 y = __builtin_amdgcn_perm(a0 >> 6, a0, 0x0c040c00u);
    const u32 z = __builtin_amdgcn_bitop3_b32(y, y << 5, 0x07070707u, 0xa8);
    const u32 ex = __builtin_amdgcn_perm(__builtin_amdgcn_perm(th, tl, z), td, t[i].x);
    const u32 nb = (i & 1) ? __builtin_amdgcn_alignbit(f.n[i >> 1], f.n[i >> 1], 4) : f.n[i >> 1];
    o[i] = (int)__builtin_amdgcn_bitop3_b32(nb, 0x87878787u, ex, 0xea);
    if (i < 7) win >>= t[i].y;
  }
  return o;
}

__global__ void __launch_bounds__(512, 2) ag_gemm(Params p) {
  __shared__ __attribute__((aligned(16))) char smem_[L_BYTES];
  LDS char* const smem = (LDS char*)smem_;
  const int lane = threadIdx.x & 63;
  const int w = __builtin_amdgcn_readfirstlane(threadIdx.x >> 6);
  const int b = blockIdx.x, slot = b & 7, q = b >> 3, grp = q >> 3, panel = q & 7;
  const int gq = lane >> 4, r16 = lane & 15;
  STAMP_INIT();
  STAMP(0);
  asm volatile("" :: "s"(p.ws[0]), "s"(p.ws[1]), "s"(p.ws[2]), "s"(p.ws[3]), "s"(p.ws[4]), "s"(p.ws[5]), "s"(p.ws[6]), "s"(p.ws[7]),
               "s"(p.x), "s"(p.w), "s"(p.sx), "s"(p.sw), "s"(p.out), "s"(p.rank));
  char* const me = p.ws[0];
  const char* const cntp = me + OFF_CNT + b * LINE;
  u32 call;

  const bool job = panel < 4, pushcu = (q & 15) == 0;
  const int pc = slot * 2 + (q >> 4);
  const int jt = grp * 4 + panel;
  const int F = pushcu ? w * 16 + ((pc + 2 * w) & 15) : 128 + (slot * 14 + jt - 1 - (jt > 8)) * 8 + w;
  const u64 wa = (u64)p.w;
  const u32x4 wrs = {(u32)wa, (u32)(wa >> 32) & 0xffffu, (u32)(KD * NL), 0x00020000u};
  const u32 wve = (u32)(lane >> 2) * NL + panel * 64 + (((lane & 3) ^ (gq & 1)) << 4), wvo = wve ^ 32u;
  const u32 lbase = (u32)(size_t)smem;
  const u32 s01 = L_STG + w * L_STAGE;
  const u32 l4 = (u32)min(lane, (int)L2_DW - 1) * 4u;
  v8i Wf[4][4];
  const int q8 = r16 >> 1, f2 = ((q8 >> 2) & 1) | ((gq & 1) << 1);
  const u32 tra = (u32)(gq * 16 + q8) * 64 + (r16 & 1) * 8;
  auto read_stage = [&](int j, u32 off) {
#pragma unroll
    for (int c = 0; c < 4; ++c) {
      LDS char* a = smem + off + tra + ((c ^ f2) << 4);
      v2i t0 = __builtin_amdgcn_ds_read_tr8_b64_v2i32((LDS v2i*)(a));
      v2i t1 = __builtin_amdgcn_ds_read_tr8_b64_v2i32((LDS v2i*)(a + 512));
      v2i t2 = __builtin_amdgcn_ds_read_tr8_b64_v2i32((LDS v2i*)(a + 4096));
      v2i t3 = __builtin_amdgcn_ds_read_tr8_b64_v2i32((LDS v2i*)(a + 4608));
      Wf[j][c] = v8i{t0.x, t0.y, t1.x, t1.y, t2.x, t2.y, t3.x, t3.y};
    }
  };
  STAMP(22);
  if (job) {
    u32x4 x0, x1;
    const unsigned char* src = p.x + (size_t)((F >> 7) * 64 + ((F >> 3) & 3) * 16 + r16) * KD + ((((F >> 5) & 3) * 8 + (F & 7)) * 128 + gq * 16);
    char* const dst = p.ws[(w + 1) & 7];
    asm volatile("global_load_dwordx4 %0, %3, off\n\tglobal_load_dwordx4 %1, %3, off offset:64\n\ts_load_dword %2, %4, 0x0"
                 : "=&v"(x0), "=&v"(x1), "=&s"(call) : "v"(src), "s"(cntp), "s"(dst) : "memory");
    if (pushcu) {
      if (EARLYW >= 1) dma_w(lbase + s01, wve, wvo, wrs, (u32)w * 65536u);
      if (EARLYW >= 2) dma_w(lbase + s01 + 8192, wve, wvo, wrs, (u32)(w + 8) * 65536u);
    }
#pragma unroll
    for (int i = 0; i < 4; ++i) {
      u32 d, c, n;
      lut_entry(lane + 64 * i, d, c, n);
      if (!pushcu) *(LDS u32x2*)(smem + s01 + (lane + 64 * i) * 8) = u32x2{c, n};
      if (i == (w & 3) && w < 4) *(LDS u32x2*)(smem + L_LUT + (lane + 64 * i) * 8) = u32x2{d, n};
    }
    if (!pushcu) *(LDS u32*)(smem + s01 + 2048 + lane * 4) = 0u;
    if (threadIdx.x < 8) *(LDS u32*)(smem + L_CTL + threadIdx.x * 4) = 0u;
    asm volatile("s_waitcnt vmcnt(%3) lgkmcnt(0)" : "+v"(x0), "+v"(x1), "+s"(call) : "n"(8 * EARLYW) : "memory");
    if (!pushcu) asm volatile("s_waitcnt vmcnt(0)" : "+v"(x0), "+v"(x1) :: "memory");
    call += 1u;
    STAMP(14);
    LDS char* sl = smem + L_XTRA + w * FRAG;
    if (pushcu) {
      const u32 off = OFF_XP + ((call & 1u) * NR + p.rank) * XP_BYTES + lane * 16;
      st16_sys(dst, off + F * FRAG, x0); st16_sys(dst, off + F * FRAG + 1024, x1);
      *(LDS u32x4*)(sl + lane * 16) = x0;
      *(LDS u32x4*)(sl + 1024 + lane * 16) = x1;
      wait_lgkm();
      __builtin_amdgcn_s_barrier();
      asm volatile("" ::: "memory");
#pragma unroll
      for (int i = 1; i < 8; ++i) {
        const int fi = (i + w) & 7, Fi = fi * 16 + ((pc + 2 * fi) & 15);
        const u32x4 a = *(LDS u32x4*)(smem + L_XTRA + fi * FRAG + lane * 16), c = *(LDS u32x4*)(smem + L_XTRA + fi * FRAG + 1024 + lane * 16);
#pragma unroll
        for (int h = 0; h < 2; ++h) {
          const int k = 2 * i + h;
          if (k >= PW) { if (k > GOS && k < GOS + PW) wait_vm<PW>(); else wait_vm<PW - 1>(); }
          if (k == GOS && lane == 0) st4_sys(dst, OFF_GO + p.rank * LINE + pc * 4, call);
          st16_sys(dst, off + Fi * FRAG + h * 1024, h ? c : a);
        }
      }
      wait_vm<0>();
      STAMP(19);
      if (lane == 0) st4_sys(dst, OFF_PSH + p.rank * LINE + pc * 4, call);
    } else {
      const u32 base = OFF_XC + (call & 1u) * XC_BYTES;
      st16_sys(me, base + XC_RAW + F * FRAG + lane * 16, x0); st16_sys(me, base + XC_RAW + F * FRAG + 1024 + lane * 16, x1);
      Packed pk;
      const bool fit = encode(v8i{(int)x0.x, (int)x0.y, (int)x0.z, (int)x0.w, (int)x1.x, (int)x1.y, (int)x1.z, (int)x1.w}, pk, smem + s01, smem + s01 + 2048, lane);
      STAMP(24);
      if (fit) {
        *(LDS u32x2*)(sl + lane * 8) = pk.c;
        *(LDS u32*)(sl + 512 + lane * 4) = pk.l;
        st16_sys(me, base + F * PK + lane * 16, pk.n);
        const u32x4 c = *(LDS u32x4*)(sl + lane * 16);
        if (lane < (int)(PK - 1024) / 16) st16_sys(me, base + F * PK + 1024 + lane * 16, c);
      } else if (lane == 0) {
        st4_sys(me, base + F * PK + 1536, HDR_RAW);
      }
      char* wsl = p.ws[0];
#pragma unroll
      for (int i = 1; i < NR; ++i) wsl = (lane & 7) == i ? p.ws[i] : wsl;
      wsl += OFF_CPY + p.rank * 512 + ((F - 128) >> 3) * 4;
      asm volatile("s_waitcnt vmcnt(0)" : "+v"(wsl) :: "memory");
      __builtin_amdgcn_s_barrier();
      STAMP(19);
      if (w == 0 && lane < NR) asm volatile("global_store_dword %0, %1, off sc0 sc1" :: "v"(wsl), "v"(call) : "memory");
    }
  }
  {
    wait_lgkm();
    if (!pushcu || EARLYW < 1) dma_w(lbase + s01, wve, wvo, wrs, (u32)w * 65536u);
    if (!pushcu || EARLYW < 2) dma_w(lbase + s01 + 8192, wve, wvo, wrs, (u32)(w + 8) * 65536u);
    if (!job) {
      if (w < 4) { u32 d, c, n; lut_entry(threadIdx.x, d, c, n); *(LDS u32x2*)(smem + L_LUT + threadIdx.x * 8) = u32x2{d, n}; }
      if (threadIdx.x < 8) *(LDS u32*)(smem + L_CTL + threadIdx.x * 4) = 0u;
      asm volatile("s_load_dword %0, %1, 0x0\n\ts_waitcnt lgkmcnt(0)" : "=&s"(call) : "s"(cntp) : "memory");
      call += 1u;
      __builtin_amdgcn_s_barrier();
    }
    STAMP(1);
    wait_vm<8>();
    read_stage(0, s01);
    wait_lgkm();
    dma_w(lbase + s01, wve, wvo, wrs, (u32)(w + 16) * 65536u);
    wait_vm<8>();
    read_stage(1, s01 + 8192);
    wait_lgkm();
    dma_w(lbase + s01 + 8192, wve, wvo, wrs, (u32)(w + 24) * 65536u);
    wait_vm<8>();
    read_stage(2, s01);
    wait_vm<0>();
    read_stage(3, s01 + 8192);
  }
  call = __builtin_amdgcn_readfirstlane(call);
  const u32 par = call & 1u;
  STAMP(2);
  const int sh = (p.rank + slot) & 7;
  char* const tmo = me + OFF_TMO;
  if (w == 1) {
    u32 xcc;
    asm volatile("s_getreg_b32 %0, hwreg(HW_REG_XCC_ID)" : "=s"(xcc));
    const u32 eo = ((xcc & 7u) * 8 + slot) * LINE;
    u32 cl = ld_agent(me + OFF_CLEAN + eo);
    if (cl < call) {
      u32 old = 0;
      if (lane == 0) old = __hip_atomic_fetch_max((GLB u32*)(me + OFF_ELECT + eo), call, __ATOMIC_RELAXED, __HIP_MEMORY_SCOPE_AGENT);
      old = __builtin_amdgcn_readfirstlane(old);
      if (old < call) {
        const char* fl = me + OFF_CPY + sh * 512;
        cl = poll_all(fl + lane * 4, fl + 256 + min(lane, 47) * 4, call, tmo, call) != ~0u ? call : ~0u;
        STAMP(20);
        __builtin_amdgcn_fence(__ATOMIC_ACQUIRE, "");
        wait_vm<0>();
        if (lane == 0) st_agent(me + OFF_CLEAN + eo, cl);
        STAMP(21);
      } else {
        cl = poll_all(me + OFF_CLEAN + eo, nullptr, call, tmo, call);
      }
    }
    STAMP(23);
    if (poll_all(me + OFF_GO + sh * LINE + (lane & 15) * 4, nullptr, call, tmo, call) == ~0u) cl = ~0u;
    if (lane == 0) {
      if (cl != call) *(LDS u32*)(smem + L_CTL + 4) = 1u;
      __hip_atomic_store((LDS u32*)(smem + L_CTL + 20), 1u, __ATOMIC_RELAXED, __HIP_MEMORY_SCOPE_WORKGROUP);
      st_agent(me + OFF_CNT + b * LINE, call);
    }
  } else {
    while (lds_ld(smem + L_CTL + 20) == 0u) __builtin_amdgcn_s_sleep(1);
  }
  asm volatile("" ::: "memory");
  STAMP(3);

  const auto rrs = __builtin_amdgcn_make_buffer_rsrc(p.ws[slot] + OFF_XC + par * XC_BYTES, 0, XC_BYTES, 0x00020000);
  const auto ors = __builtin_amdgcn_make_buffer_rsrc(p.out, 0, 4096 * NL * 4, 0x00020000);
  const u32 fi32 = (u32)(grp * 8 + w);
  const u32 vn = fi32 * PK + lane * 16, vc = fi32 * PK + 1024 + lane * 8, vl = fi32 * PK + 1536 + l4;
  const u32 vr = fi32 * FRAG + lane * 16;
  const float scale = *(const CST float*)p.sx * *(const CST float*)p.sw;
  struct Frag { u32x4 a, b; u32 l; };
  auto ld = [&](Frag& f, auto rs, u32 va, u32 vb, u32 so, auto aux, u32 vv) {
    f.a = __builtin_amdgcn_raw_buffer_load_b128(rs, va, so, aux);
    f.b = __builtin_amdgcn_raw_buffer_load_b128(rs, vb, so, aux);
    f.l = __builtin_amdgcn_raw_buffer_load_b32(rs, vv, so, aux);
  };
  auto unpack = [&](const Frag& f, auto rs, u32 rawso, bool raw, auto aux) {
    const u32 hdr = raw ? HDR_RAW : __builtin_amdgcn_readfirstlane(f.l);
    v8i o;
    if (__builtin_expect(!(hdr & HDR_RAW), 1)) {
      o = decode(Packed{f.a, u32x2{f.b.x, f.b.y}, f.l}, hdr, smem + L_LUT);
    } else {
      u32x4 a = f.a, c = f.b;
      if (!raw) {
        a = __builtin_amdgcn_raw_buffer_load_b128(rs, vr, rawso, aux);
        c = __builtin_amdgcn_raw_buffer_load_b128(rs, vr + 1024, rawso, aux);
        asm volatile("" : "+v"(a), "+v"(c));
      }
      o = v8i{(int)a.x, (int)a.y, (int)a.z, (int)a.w, (int)c.x, (int)c.y, (int)c.z, (int)c.w};
    }
    return o;
  };
  const std::integral_constant<int, 0> PLAIN{};
  auto put_partial = [&](const v4f* acc, int buf) {
#pragma unroll
    for (int c = 0; c < 4; ++c) *(LDS v4f*)(smem + s01 + buf * PBUF + r16 * PROW + c * 64 + gq * 16) = acc[c];
    wait_lgkm();
    if (lane == 0) __hip_atomic_fetch_add((LDS u32*)(smem + L_CTL + 8 + buf * 4), 1u, __ATOMIC_RELAXED, __HIP_MEMORY_SCOPE_WORKGROUP);
  };
  auto reduce = [&](int row0, int t, bool last) {
    const int buf = t & 1;
    while (lds_ld(smem + L_CTL + 8 + buf * 4) < 8u * (u32)((t >> 1) + 1)) __builtin_amdgcn_s_sleep(1);
    asm volatile("" ::: "memory");
    const int m = 2 * w + (lane >> 5);
    LDS char* a = smem + L_STG + buf * PBUF + m * PROW + (lane & 31) * 8;
    v2f s = *(LDS v2f*)a;
#pragma unroll
    for (int i = 1; i < 8; ++i) s += *(LDS v2f*)(a + i * L_STAGE);
    const bool bad = lds_ld(smem + L_CTL + 4) != 0u;
    u32 o[2];
#pragma unroll
    for (int i = 0; i < 2; ++i) {
      const float y = s[i] * scale;
      const float z = fminf(fmaxf(y, -60.f), 60.f);
      o[i] = __builtin_bit_cast(u32, bad ? __builtin_nanf("") : y * __builtin_amdgcn_rcpf(1.f + __expf(-z)));
    }
    const u32x4 v = {o[0], o[1], (u32)__builtin_amdgcn_mov_dpp((int)o[0], 0xb1, 0xf, 0xf, true), (u32)__builtin_amdgcn_mov_dpp((int)o[1], 0xb1, 0xf, 0xf, true)};
    if ((lane & 1) == 0) {
      const u32 off = ((u32)(row0 + grp * 16 + m) * NL + panel * 64 + (lane & 31) * 2) * 4u;
      if (last) __builtin_amdgcn_raw_buffer_store_b128(v, ors, off, 0, LASTAUX); else __builtin_amdgcn_raw_buffer_store_b128(v, ors, off, 0, OUTAUX);
    }
  };

  if (slot == 0) {
    const auto lrs = __builtin_amdgcn_make_buffer_rsrc(me, 0, WS_USED, 0x00020000);
    u32x4 nx[8];
    auto ldt = [&](int t) {
      const u32 so = t < 7 ? OFF_XC + par * XC_BYTES + XC_RAW + (u32)(t + 1) * (128 * FRAG) : OFF_XP + (par * NR + ((p.rank + t - 7) & 7)) * XP_BYTES;
#pragma unroll
      for (int i = 0; i < 8; ++i) nx[i] = __builtin_amdgcn_raw_buffer_load_b128(lrs, vr + (i & 1) * 1024, so + (i >> 1) * (32 * FRAG), 17);
    };
    auto row0 = [&](int t) { return t < 7 ? p.rank * 512 + (t + 1) * 64 : ((p.rank + t - 7) & 7) * 512; };
    ldt(0);
    for (int t = 0; t < 15; ++t) {
      u32x4 cu[8];
#pragma unroll
      for (int i = 0; i < 8; ++i) { cu[i] = nx[i]; asm volatile("" : "+v"(cu[i])); }
      if (t == 6) {
        const char* fl = me + OFF_PSH + gq * LINE + r16 * 4;
        if (poll_all(fl, fl + 4 * LINE, call, tmo, call) == ~0u && lane == 0) *(LDS u32*)(smem + L_CTL + 4) = 1u;
      }
      asm volatile("" ::: "memory");
      if (t < 14) ldt(t + 1);
      asm volatile("" ::: "memory");
      __builtin_amdgcn_sched_barrier(0);
      v4f acc[4] = {};
#pragma unroll
      for (int j = 0; j < 4; ++j) {
        const u32x4 a = cu[2 * j], c2 = cu[2 * j + 1];
        const v8i xv = {(int)a.x, (int)a.y, (int)a.z, (int)a.w, (int)c2.x, (int)c2.y, (int)c2.z, (int)c2.w};
#pragma unroll
        for (int c = 0; c < 4; ++c) acc[c] = __builtin_amdgcn_mfma_scale_f32_16x16x128_f8f6f4(Wf[j][c], xv, acc[c], 0, 0, 0, 0, 0, 0);
      }
      if (t > 0) reduce(row0(t - 1), t - 1, false);
      put_partial(acc, t & 1);
      if (t < 7) STAMP(5 + t);
    }
    reduce(row0(14), 14, false);
  } else {
    Frag R[DEPTH];
    bool rraw = false, fraw[DEPTH];
    u32 ra = vn, rb = vc, rstep = STEP, rso = 4 * STEP;
#pragma unroll
    for (int i = 0; i < DEPTH; ++i) { ld(R[i], rrs, ra, rb, rso, PLAIN, vl); fraw[i] = false; rso += STEP; asm volatile("" ::: "memory"); __builtin_amdgcn_sched_barrier(0); }
    const u32 vt = lane < TOUCH ? fi32 * PK + lane * 64 : 0x80000000u;
    u32 pf = TOUCH ? __builtin_amdgcn_raw_buffer_load_b32(rrs, vt, rso, 0) : 0u;
    for (int r = 1; r < 8; ++r) {
      v4f acc[4] = {};
#pragma unroll
      for (int j = 0; j < 4; ++j) {
        __builtin_amdgcn_sched_barrier(0);
        if (PRIO) __builtin_amdgcn_s_setprio(3);
        Frag cur = R[j % DEPTH];
        const bool craw = fraw[j % DEPTH];
        asm volatile("" : "+v"(cur.a), "+v"(cur.b), "+v"(cur.l));
        if (!rraw && (__builtin_amdgcn_readfirstlane(cur.l) & HDR_RAW)) {
          rraw = true; ra = vr; rb = vr + 1024; rstep = 32 * FRAG; rso = XC_RAW + (u32)(r * 4 + j + DEPTH) * (32 * FRAG);
        }
        if (r == SR) STAMP(25 + j);
        ld(R[j % DEPTH], rrs, r * 4 + j + DEPTH < 32 ? ra : 0x80000000u, r * 4 + j + DEPTH < 32 ? rb : 0x80000000u, rso, PLAIN, r * 4 + j + DEPTH < 32 ? vl : 0x80000000u);
        fraw[j % DEPTH] = rraw;
        rso += rstep;
        if (TOUCH) {
          asm volatile("" :: "v"(pf));
          pf = __builtin_amdgcn_raw_buffer_load_b32(rrs, r * 4 + j + DEPTH < 31 ? vt : 0x80000000u, rso, 0);
        }
        asm volatile("" ::: "memory");
        if (PRIO) __builtin_amdgcn_s_setprio(0);
        __builtin_amdgcn_sched_barrier(0);
        if (RPOS == 0 && j == 1 && r > 1) reduce(sh * 512 + (r - 1) * 64, r - 2, false);
        if (r == SR && j == 2) STAMP(15);
        v8i xv = unpack(cur, rrs, XC_RAW + (u32)(r * 4 + j) * (32 * FRAG), craw, PLAIN);
        if (r == SR && j == 2) STAMP(16);
#pragma unroll
        for (int c = 0; c < 4; ++c) acc[c] = __builtin_amdgcn_mfma_scale_f32_16x16x128_f8f6f4(Wf[j][c], xv, acc[c], 0, 0, 0, 0, 0, 0);
        if (RPOS == 1 && j == 0 && r > 1) reduce(sh * 512 + (r - 1) * 64, r - 2, false);
      }
      put_partial(acc, (r - 1) & 1);
      STAMP(4 + r);
    }
    reduce(sh * 512 + 7 * 64, 6, true);
  }
  STAMP(12);
  STAMP(13);
  STAMP_FLUSH();
}

extern "C" void kernel_launch(int rank, int nranks, ncclComm_t comm,
                              hipStream_t stream,
                              void* const* d_in, const int* in_sizes, int n_in,
                              void* d_out, int out_size,
                              const int* in_dtypes, int out_dtype,
                              void* const* d_out_all,
                              void* const* d_ws_all, size_t ws_bytes) {
  if (nranks != NR || (unsigned)rank >= (unsigned)NR || n_in != 4 || ws_bytes < (size_t)WS_USED + (1u << 20) ||
      in_sizes[0] != 512 * KD || in_sizes[1] != KD * NL || out_size != 4096 * NL) {
    fprintf(stderr, "unexpected problem shape\n");
    return;
  }
  Params p;
  for (int i = 0; i < NR; ++i) p.ws[i] = (char*)d_ws_all[(rank + i) & 7];
  p.x = (const unsigned char*)d_in[0];
  p.w = (const unsigned char*)d_in[1];
  p.sx = (const float*)d_in[2];
  p.sw = (const float*)d_in[3];
  p.out = (float*)d_out;
  p.rank = rank;
  p.pad = 0;
  ag_gemm<<<256, 512, 0, stream>>>(p);
  HIP_CHECK(hipGetLastError());
}
```

```cpp
#include <hip/hip_runtime.h>
#include <rccl/rccl.h>
#include <cstdio>

#define HIP_CHECK(expr) do { hipError_t _e = (expr); if (_e != hipSuccess) { \
    fprintf(stderr, "HIP error %s at %s:%d\n", hipGetErrorString(_e), __FILE__, __LINE__); \
    return; } } while (0)

typedef unsigned u32;
typedef unsigned long long u64;
typedef u32 u32x4 __attribute__((ext_vector_type(4)));
typedef u32 u32x2 __attribute__((ext_vector_type(2)));
typedef int v2i __attribute__((ext_vector_type(2)));
typedef int v4i __attribute__((ext_vector_type(4)));
typedef int v8i __attribute__((ext_vector_type(8)));
typedef float v4f __attribute__((ext_vector_type(4)));
typedef float v2f __attribute__((ext_vector_type(2)));
#define LDS __attribute__((address_space(3)))
#define GLB __attribute__((address_space(1)))
#define CST __attribute__((address_space(4)))

constexpr int NR = 8;
constexpr int KD = 4096, NL = 512;
constexpr u32 FRAG = 2048;
constexpr u32 LINE = 128;
constexpr u32 OFF_CNT = 0;
constexpr u32 OFF_TMO = OFF_CNT + 256 * LINE;
constexpr u32 OFF_ELECT = OFF_TMO + LINE;
constexpr u32 OFF_CLEAN = OFF_ELECT + 64 * LINE;
constexpr u32 OFF_CPY = OFF_CLEAN + 64 * LINE;
constexpr u32 CPY_GO = 112 * 4;
constexpr u32 OFF_PSH = OFF_CPY + NR * 512;
constexpr u32 L2_DW = 63;
constexpr u32 BLK_C = 8 * 1024, BLK_L = 8 * 1536;
constexpr u32 BLK = BLK_L + 2048;
constexpr u32 STEP = 4 * BLK;
constexpr u32 OFF_XC = 1u << 20;
constexpr u32 XC_RAW = 2u << 20, XC_BYTES = XC_RAW + 1024 * FRAG;
constexpr u32 OFF_XP = OFF_XC + 2 * XC_BYTES;
constexpr u32 XP_BYTES = 128 * FRAG;
constexpr u32 WS_USED = OFF_XP + 2 * NR * XP_BYTES;
static_assert(32 * STEP <= XC_RAW && STEP % 128 == 0, "layout");
#define STAMP(i) do {} while (0)
#define STAMP_INIT() do {} while (0)
#define STAMP_FLUSH() do {} while (0)
#define L_STAMP 0
constexpr u32 L_LUT = 0;
constexpr u32 L_STG = 2048, L_STAGE = 16384;
constexpr u32 L_XTRA = L_STG + 8 * L_STAGE;
constexpr u32 L_CTL = L_XTRA + 8 * FRAG;
constexpr u32 L_BYTES = L_CTL + 64 + L_STAMP;
constexpr u32 PROW = 272;
constexpr u32 PBUF = 16 * PROW;
#define PW 3
#define GOS 9
#define TOUCH 2
#define HOLDW 24
#define SR 6
#define WPF 1
#define DRIVE 8
#define DWIN 16
#define HOLD 0
constexpr u64 DEADLINE_TICKS = 50000000ull;

struct Params {
  char* ws[NR];
  const unsigned char* x;
  const unsigned char* w;
  const float* sx;
  const float* sw;
  float* out;
  int rank;
  int pad;
};

template <int N> __device__ __forceinline__ void wait_vm() { asm volatile("s_waitcnt vmcnt(%0)" :: "n"(N) : "memory"); }
__device__ __forceinline__ void wait_lgkm() { asm volatile("s_waitcnt lgkmcnt(0)" ::: "memory"); }
__device__ __forceinline__ u32 ld_sys(const char* a) { return __hip_atomic_load((GLB u32*)a, __ATOMIC_RELAXED, __HIP_MEMORY_SCOPE_SYSTEM); }
__device__ __forceinline__ u32 ld_agent(const char* a) { return __hip_atomic_load((GLB u32*)a, __ATOMIC_RELAXED, __HIP_MEMORY_SCOPE_AGENT); }
__device__ __forceinline__ void st_agent(char* a, u32 v) { __hip_atomic_store((GLB u32*)a, v, __ATOMIC_RELAXED, __HIP_MEMORY_SCOPE_AGENT); }
__device__ __forceinline__ u32 lds_ld(LDS char* a) { return __hip_atomic_load((LDS u32*)a, __ATOMIC_RELAXED, __HIP_MEMORY_SCOPE_WORKGROUP); }

__device__ __forceinline__ void dma_w(u32 lds, u32 ve, u32 vo, u32x4 rs, u32 so) {
  u32 keep;
#define DMA_STEP(V) "s_add_u32 m0, m0, 0x400\n\ts_add_u32 %1, %1, 0x2000\n\tbuffer_load_dwordx4 " V ", %5, %1 offen lds\n\t"
  asm volatile("s_mov_b32 %0, m0\n\ts_mov_b32 m0, %2\n\ts_nop 0\n\tbuffer_load_dwordx4 %3, %5, %1 offen lds\n\t"
               DMA_STEP("%4") DMA_STEP("%3") DMA_STEP("%4") DMA_STEP("%3") DMA_STEP("%4") DMA_STEP("%3") DMA_STEP("%4")
               "s_mov_b32 m0, %0"
               : "=&s"(keep), "+s"(so) : "s"(lds), "v"(ve), "v"(vo), "s"(rs) : "memory", "scc");
#undef DMA_STEP
}
__device__ __forceinline__ void st16_sys(char* base, u32 off, u32x4 v) {
  asm volatile("global_store_dwordx4 %0, %1, %2 sc0 sc1\n\ts_nop 1" :: "v"(off), "v"(v), "s"(base) : "memory");
}
__device__ __forceinline__ void st8_sys(char* base, u32 off, u32x2 v) {
  asm volatile("global_store_dwordx2 %0, %1, %2 sc0 sc1\n\ts_nop 1" :: "v"(off), "v"(v), "s"(base) : "memory");
}
__device__ __forceinline__ void st4_sys(char* base, u32 off, u32 v) {
  asm volatile("global_store_dword %0, %1, %2 sc0 sc1\n\ts_nop 1" :: "v"(off), "v"(v), "s"(base) : "memory");
}
__device__ __forceinline__ u32 poll_all(const char* word, const char* word2, u32 target, char* tmo, u32 call) {
  const u64 t0 = __builtin_amdgcn_s_memrealtime();
  for (u32 it = 1;; ++it) {
    const u32 v = ld_sys(word), v2 = word2 ? ld_sys(word2) : v;
    if (__all(v >= target && v2 >= target)) return __builtin_amdgcn_readfirstlane(v);
    if ((it & 15u) == 0u && (ld_agent(tmo) == call || __builtin_amdgcn_s_memrealtime() - t0 > DEADLINE_TICKS)) { st_agent(tmo, call); return ~0u; }
    __builtin_amdgcn_s_sleep(1);
  }
}

constexpr u32 L2_CAP = (L2_DW * 32 - 8) / 3;
constexpr u32 IDX_RAW = 0x8000;

__device__ __forceinline__ u32 wave_scan(u32 v) {
  v += __builtin_amdgcn_update_dpp(0u, v, 0x111, 0xf, 0xf, false);
  v += __builtin_amdgcn_update_dpp(0u, v, 0x112, 0xf, 0xf, false);
  v += __builtin_amdgcn_update_dpp(0u, v, 0x114, 0xf, 0xf, false);
  v += __builtin_amdgcn_update_dpp(0u, v, 0x118, 0xf, 0xf, false);
  v += __builtin_amdgcn_update_dpp(0u, v, 0x142, 0xa, 0xf, false);
  v += __builtin_amdgcn_update_dpp(0u, v, 0x143, 0xc, 0xf, false);
  return v;
}
__device__ __forceinline__ void lut_entry(u32 cb, u32& dsel, u32& csel, u32& s3) {
  u32 n = 0; dsel = 0; csel = 0x0c0c0c0cu;
#pragma unroll
  for (int b = 0; b < 4; ++b) {
    const u32 c = (cb >> (2 * b)) & 3u;
    const bool e = c == 3u;
    dsel |= (e ? 4u + n : c) << (8 * b);
    csel = e ? (csel & ~(0xffu << (8 * n))) | ((u32)b << (8 * n)) : csel;
    n += e;
  }
  s3 = 3u * n;
}
struct Packed { u32x4 n; u32x2 c; u32 l; };

template <int B> __device__ __forceinline__ u32 byte_shl(u32 c, u32 sh) {
  u32 r;
  if (B == 0) asm("v_lshlrev_b32_sdwa %0, %1, %2 dst_sel:DWORD dst_unused:UNUSED_PAD src0_sel:DWORD src1_sel:BYTE_0" : "=v"(r) : "v"(sh), "v"(c));
  if (B == 1) asm("v_lshlrev_b32_sdwa %0, %1, %2 dst_sel:DWORD dst_unused:UNUSED_PAD src0_sel:DWORD src1_sel:BYTE_1" : "=v"(r) : "v"(sh), "v"(c));
  if (B == 2) asm("v_lshlrev_b32_sdwa %0, %1, %2 dst_sel:DWORD dst_unused:UNUSED_PAD src0_sel:DWORD src1_sel:BYTE_2" : "=v"(r) : "v"(sh), "v"(c));
  if (B == 3) asm("v_lshlrev_b32_sdwa %0, %1, %2 dst_sel:DWORD dst_unused:UNUSED_PAD src0_sel:DWORD src1_sel:BYTE_3" : "=v"(r) : "v"(sh), "v"(c));
  return r;
}
__device__ __forceinline__ u32 encode(const v8i& xi, Packed& o, LDS char* lutc, LDS char* buf, int lane) {
  u32 x[8];
#pragma unroll
  for (int i = 0; i < 8; ++i) x[i] = (u32)xi[i];
  u32 se = 0;
#pragma unroll
  for (int i = 0; i < 8; i += 2) se = __builtin_amdgcn_sad_u8(x[i] & 0x78787878u, 0u, se);
  const u32 tot = __builtin_amdgcn_readlane(wave_scan(se), 63);
  const u32 e0 = min((tot < 256u ? 0u : tot - 256u) >> 13, 10u);
  const u32 bias = (13u - e0) * 0x01010101u;
  u32 inv1 = 0, inv2 = 0, ca[8], kb[8];
#pragma unroll
  for (int i = 0; i < 8; ++i) {
    const u32 gb = ((x[i] >> 3) & 0x0f0f0f0fu) + bias;
    const u32 sel = gb & 0x07070707u;
    const u32 a = __builtin_amdgcn_perm(0x1008001cu, 0x1b1a1918u, sel);
    const u32 b = __builtin_amdgcn_perm(0x98989898u, 0x981f1e1du, sel);
    const u32 m = __builtin_amdgcn_perm(0u, 0u, __builtin_amdgcn_bitop3_b32(gb >> 4, 0x01010101u, 0x0c0c0c0cu, 0xea));
    const u32 code = __builtin_amdgcn_bitop3_b32(m, b, a, 0xca);
    inv1 = __builtin_amdgcn_bitop3_b32(inv1, gb, gb >> 1, 0xf9);
    inv2 |= code;
    ca[i] = __builtin_amdgcn_udot4(code & 0x18181818u, 0x40100401u, 0u, false);
    kb[i] = code & 0x07070707u;
  }
  u64 acc = 0; u32 n3 = 0;
#pragma unroll
  for (int i = 7; i >= 0; --i) {
    const u32x2 t = *(LDS u32x2*)(lutc + ca[i]);
    const u32 comp = __builtin_amdgcn_perm(kb[i], kb[i], t.x);
    acc = (acc << t.y) | __builtin_amdgcn_udot4(comp, 0x00400801u, byte_shl<3>(comp, 9u), false);
    n3 += t.y;
  }
  const u32 incl = wave_scan(n3);
  const u32 all3 = __builtin_amdgcn_readlane(incl, 63);
  const bool fit = !__any(((inv1 & 0x08080808u) | (inv2 & 0x80808080u)) != 0u || n3 > 63u) && all3 <= 3u * L2_CAP;
  o.c = u32x2{(ca[0] >> 3) | (ca[1] << 5) | (ca[2] << 13) | (ca[3] << 21), (ca[4] >> 3) | (ca[5] << 5) | (ca[6] << 13) | (ca[7] << 21)};
#pragma unroll
  for (int j = 0; j < 4; ++j) {
    const u32 od = x[2 * j + 1] & 0x87878787u;
    o.n[j] = (x[2 * j] & 0x87878787u) | __builtin_amdgcn_alignbit(od, od, 28);
  }
  o.l = 0;
  if (!fit) return 0u;
  const u32 off = incl - n3 + 8u, sh = off & 31u;
  const u64 lo = acc << sh;
  const u32 hi = ((u32)(acc >> 32) >> 1) >> (31u - sh);
  LDS u32* d = (LDS u32*)(buf + ((off >> 5) << 2));
  __hip_atomic_fetch_or(d, (u32)lo | (lane == 0 ? e0 : 0u), __ATOMIC_RELAXED, __HIP_MEMORY_SCOPE_WAVEFRONT);
  __hip_atomic_fetch_or(d + 1, (u32)(lo >> 32), __ATOMIC_RELAXED, __HIP_MEMORY_SCOPE_WAVEFRONT);
  __hip_atomic_fetch_or(d + 2, hi, __ATOMIC_RELAXED, __HIP_MEMORY_SCOPE_WAVEFRONT);
  o.l = __hip_atomic_load((LDS u32*)(buf + lane * 4), __ATOMIC_RELAXED, __HIP_MEMORY_SCOPE_WAVEFRONT);
  return (all3 + 8u + 31u) >> 5;
}

__device__ __forceinline__ v8i exponents(u32x2 c, u32 l, LDS char* lutd) {
  u32x2 t[8];
  t[0] = *(LDS u32x2*)(lutd + byte_shl<0>(c.x, 3u)); t[1] = *(LDS u32x2*)(lutd + byte_shl<1>(c.x, 3u));
  t[2] = *(LDS u32x2*)(lutd + byte_shl<2>(c.x, 3u)); t[3] = *(LDS u32x2*)(lutd + byte_shl<3>(c.x, 3u));
  t[4] = *(LDS u32x2*)(lutd + byte_shl<0>(c.y, 3u)); t[5] = *(LDS u32x2*)(lutd + byte_shl<1>(c.y, 3u));
  t[6] = *(LDS u32x2*)(lutd + byte_shl<2>(c.y, 3u)); t[7] = *(LDS u32x2*)(lutd + byte_shl<3>(c.y, 3u));
  __builtin_amdgcn_sched_barrier(0);
  const u32 n = __builtin_popcount(__builtin_amdgcn_bitop3_b32(c.x, c.x >> 1, 0x55555555u, 0x80)) +
                __builtin_popcount(__builtin_amdgcn_bitop3_b32(c.y, c.y >> 1, 0x55555555u, 0x80));
  u32 incl = wave_scan(n);
  asm("" : "+v"(incl));
  const u32 off = __umul24(incl - n, 3u) + 8u;
  const int a = (int)(off >> 3);
  const u32 w0 = __builtin_amdgcn_ds_bpermute(a, l), w1 = __builtin_amdgcn_ds_bpermute(a + 4, l), w2 = __builtin_amdgcn_ds_bpermute(a + 8, l);
  u64 win = ((u64)__builtin_amdgcn_alignbit(w2, w1, off) << 32) | __builtin_amdgcn_alignbit(w1, w0, off);
  const u32 e8 = (__builtin_amdgcn_readfirstlane(l) & 15u) * 0x08080808u;
  const u32 td = 0x00100800u + e8, tl = (0x70686058u + e8) & 0x7f7f7f7fu, th = (0xa8a09878u + e8) & 0x7f7f7f7fu;
  v8i o;
#pragma unroll
  for (int i = 0; i < 8; ++i) {
    const u32 a0 = (u32)win;
    const u32 y = __builtin_amdgcn_perm(a0 >> 6, a0, 0x0c040c00u);
    const u32 z = __builtin_amdgcn_bitop3_b32(y, y << 5, 0x07070707u, 0xa8);
    o[i] = (int)__builtin_amdgcn_perm(__builtin_amdgcn_perm(th, tl, z), td, t[i].x);
    if (i < 7) win >>= t[i].y;
  }
  return o;
}
__device__ __forceinline__ v8i merge(u32x4 n, v8i ex) {
#pragma unroll
  for (int i = 0; i < 8; ++i) {
    const u32 nb = (i & 1) ? __builtin_amdgcn_alignbit(n[i >> 1], n[i >> 1], 4) : n[i >> 1];
    ex[i] = (int)__builtin_amdgcn_bitop3_b32(nb, 0x87878787u, (u32)ex[i], 0xea);
  }
  return ex;
}

__global__ void __launch_bounds__(512, 2) ag_gemm(Params p) {
  __shared__ __attribute__((aligned(16))) char smem_[L_BYTES];
  LDS char* const smem = (LDS char*)smem_;
  const int lane = threadIdx.x & 63;
  const int w = __builtin_amdgcn_readfirstlane(threadIdx.x >> 6);
  const int b = blockIdx.x, slot = b & 7, q = b >> 3, grp = q >> 3, panel = q & 7;
  const int gq = lane >> 4, r16 = lane & 15;
  STAMP_INIT();
  STAMP(0);
  asm volatile("" :: "s"(p.ws[0]), "s"(p.ws[1]), "s"(p.ws[2]), "s"(p.ws[3]), "s"(p.ws[4]), "s"(p.ws[5]), "s"(p.ws[6]), "s"(p.ws[7]),
               "s"(p.x), "s"(p.w), "s"(p.sx), "s"(p.sw), "s"(p.out), "s"(p.rank));
  char* const me = p.ws[0];
  const char* const cntp = me + OFF_CNT + b * LINE;
  u32 call;

  const bool job = panel < 4, pushcu = (q & 15) == 0;
  const int pc = slot * 2 + (q >> 4);
  const int jt = grp * 4 + panel;
  const int F = pushcu ? w * 16 + ((pc + 2 * w) & 15) : 128 + (slot * 14 + jt - 1 - (jt > 8)) * 8 + w;
  const u64 wa = (u64)p.w;
  const u32x4 wrs = {(u32)wa, (u32)(wa >> 32) & 0xffffu, (u32)(KD * NL), 0x00020000u};
  const u32 wve = (u32)(lane >> 2) * NL + panel * 64 + (((lane & 3) ^ (gq & 1)) << 4), wvo = wve ^ 32u;
  const u32 lbase = (u32)(size_t)smem;
  const u32 s01 = L_STG + w * L_STAGE;
  v8i Wf[4][4];
  const int q8 = r16 >> 1, f2 = ((q8 >> 2) & 1) | ((gq & 1) << 1);
  const u32 tra = (u32)(gq * 16 + q8) * 64 + (r16 & 1) * 8;
  auto read_stage = [&](int j, u32 off) {
#pragma unroll
    for (int c = 0; c < 4; ++c) {
      LDS char* a = smem + off + tra + ((c ^ f2) << 4);
      v2i t0 = __builtin_amdgcn_ds_read_tr8_b64_v2i32((LDS v2i*)(a));
      v2i t1 = __builtin_amdgcn_ds_read_tr8_b64_v2i32((LDS v2i*)(a + 512));
      v2i t2 = __builtin_amdgcn_ds_read_tr8_b64_v2i32((LDS v2i*)(a + 4096));
      v2i t3 = __builtin_amdgcn_ds_read_tr8_b64_v2i32((LDS v2i*)(a + 4608));
      Wf[j][c] = v8i{t0.x, t0.y, t1.x, t1.y, t2.x, t2.y, t3.x, t3.y};
    }
  };
  const int sh = (p.rank + slot) & 7;
  char* const tmo = me + OFF_TMO;
  u32 xcc;
  asm volatile("s_getreg_b32 %0, hwreg(HW_REG_XCC_ID)" : "=s"(xcc));
  const u32 eo = ((xcc & 7u) * 8 + slot) * LINE;
  auto lead = [&]() {
    u32 old = 0;
    if (lane == 0) old = __hip_atomic_fetch_max((GLB u32*)(me + OFF_ELECT + eo), call, __ATOMIC_RELAXED, __HIP_MEMORY_SCOPE_AGENT);
    if (__builtin_amdgcn_readfirstlane(old) >= call) return 0u;
    const char* fl = me + OFF_CPY + sh * 512;
    const u32 cl = poll_all(fl + lane * 4, fl + 256 + lane * 4, call, tmo, call) != ~0u ? call : ~0u;
    STAMP(20);
    __builtin_amdgcn_fence(__ATOMIC_ACQUIRE, "");
    wait_vm<0>();
    STAMP(21);
    if (lane == 0) st_agent(me + OFF_CLEAN + eo, cl);
    return cl;
  };
  STAMP(22);
  if (HOLD && !pushcu) __builtin_amdgcn_s_sleep(HOLD);
  if (job) {
    u32x4 x0, x1;
    const unsigned char* src = p.x + (size_t)((F >> 7) * 64 + ((F >> 3) & 3) * 16 + r16) * KD + ((((F >> 5) & 3) * 8 + (F & 7)) * 128 + gq * 16);
    char* const dst = p.ws[(w + 1) & 7];
    asm volatile("global_load_dwordx4 %0, %3, off\n\tglobal_load_dwordx4 %1, %3, off offset:64\n\ts_load_dword %2, %4, 0x0"
                 : "=&v"(x0), "=&v"(x1), "=&s"(call) : "v"(src), "s"(cntp), "s"(dst) : "memory");
#pragma unroll
    for (int i = 0; i < 4; ++i) {
      u32 d, c, n;
      lut_entry(lane + 64 * i, d, c, n);
      if (!pushcu) *(LDS u32x2*)(smem + s01 + (lane + 64 * i) * 8) = u32x2{c, n};
      if (i == (w & 3) && w < 4) *(LDS u32x2*)(smem + L_LUT + (lane + 64 * i) * 8) = u32x2{d, n};
    }
    if (!pushcu) *(LDS u32*)(smem + s01 + 2048 + lane * 4) = 0u;
    if (threadIdx.x < 8) *(LDS u32*)(smem + L_CTL + threadIdx.x * 4) = 0u;
    asm volatile("s_waitcnt vmcnt(0) lgkmcnt(0)" : "+v"(x0), "+v"(x1), "+s"(call) :: "memory");
    call += 1u;
    STAMP(14);
    LDS char* sl = smem + L_XTRA + w * FRAG;
    if (pushcu) {
      const u32 off = OFF_XP + ((call & 1u) * NR + p.rank) * XP_BYTES + lane * 16;
      st16_sys(dst, off + F * FRAG, x0); st16_sys(dst, off + F * FRAG + 1024, x1);
      *(LDS u32x4*)(sl + lane * 16) = x0;
      *(LDS u32x4*)(sl + 1024 + lane * 16) = x1;
      wait_lgkm();
      __builtin_amdgcn_s_barrier();
      asm volatile("" ::: "memory");
#pragma unroll
      for (int i = 1; i < 8; ++i) {
        const int fi = (i + w) & 7, Fi = fi * 16 + ((pc + 2 * fi) & 15);
        const u32x4 a = *(LDS u32x4*)(smem + L_XTRA + fi * FRAG + lane * 16), c = *(LDS u32x4*)(smem + L_XTRA + fi * FRAG + 1024 + lane * 16);
#pragma unroll
        for (int h = 0; h < 2; ++h) {
          const int k = 2 * i + h;
          if (k >= PW) { if (k > GOS && k < GOS + PW) wait_vm<PW>(); else wait_vm<PW - 1>(); }
          if (k == 6) STAMP(24);
          if (k == 10) STAMP(20);
          if (k == 14) STAMP(21);
          if (k == GOS && lane == 0) st4_sys(dst, OFF_CPY + p.rank * 512 + CPY_GO + pc * 4, call);
          st16_sys(dst, off + Fi * FRAG + h * 1024, h ? c : a);
        }
      }
      wait_vm<0>();
      STAMP(19);
      if (lane == 0) st4_sys(dst, OFF_PSH + p.rank * LINE + pc * 4, call);
    } else {
      const u32 base = OFF_XC + (call & 1u) * XC_BYTES;
      st16_sys(me, base + XC_RAW + F * FRAG + lane * 16, x0); st16_sys(me, base + XC_RAW + F * FRAG + 1024 + lane * 16, x1);
      Packed pk;
      const u32 len = encode(v8i{(int)x0.x, (int)x0.y, (int)x0.z, (int)x0.w, (int)x1.x, (int)x1.y, (int)x1.z, (int)x1.w}, pk, smem + s01, smem + s01 + 2048, lane);
      STAMP(24);
      const u32 blk = base + (u32)(F >> 3) * BLK;
      if (lane == 0) *(LDS u32*)(smem + L_CTL + 32 + w * 4) = len;
      if (len) {
        *(LDS u32x2*)(sl + lane * 8) = pk.c;
        st16_sys(me, blk + w * 1024 + lane * 16, pk.n);
        const u32x4 c = *(LDS u32x4*)(sl + lane * 16);
        if (lane < 32) st16_sys(me, blk + BLK_C + w * 512 + lane * 16, c);
      }
      wait_lgkm();
      __builtin_amdgcn_s_barrier();
      asm volatile("" ::: "memory");
      const u32 ln = lane < 8 ? *(LDS u32*)(smem + L_CTL + 32 + lane * 4) : 0u, st = wave_scan(ln) - ln;
      if (lane < (int)len) st4_sys(me, blk + BLK_L + ((u32)__builtin_amdgcn_readlane(st, w) + lane) * 4, pk.l);
      if (w == 0 && lane < 8)
        asm volatile("global_store_short %0, %1, %2 sc0 sc1" :: "v"(base + (u32)(F >> 3) * 16 + lane * 2), "v"(ln ? st | (ln << 9) : IDX_RAW), "s"(me) : "memory");
      char* wsl = p.ws[0];
#pragma unroll
      for (int i = 1; i < NR; ++i) wsl = (lane & 7) == i ? p.ws[i] : wsl;
      wsl += OFF_CPY + p.rank * 512 + ((F - 128) >> 3) * 4;
      asm volatile("s_waitcnt vmcnt(0)" : "+v"(wsl) :: "memory");
      __builtin_amdgcn_s_barrier();
      STAMP(19);
      if (w == 0) {
        if (lane < NR) asm volatile("global_store_dword %0, %1, off sc0 sc1" :: "v"(wsl), "v"(call) : "memory");
        if (jt == 1 && lead() != 0u && slot != 0 && DRIVE) {
          const u64 xa = (u64)(p.ws[slot] + base);
          const u32x4 xrs = {(u32)xa, (u32)(xa >> 32) & 0xffffu, XC_RAW, 0x00020000u};
          u32 keep;
          STAMP(23);
          for (int i = -1; i < DRIVE * 4; ++i) {
            const u32 vo = i < 0 ? (lane < 32 ? lane * 64 : 0x80000000u)
                                 : ((i & 3) < 3 || lane < 22 ? 4 * STEP + (u32)(i >> 2) * BLK + (i & 3) * 4096 + lane * 64 : 0x80000000u);
            asm volatile("s_mov_b32 %0, m0\n\ts_mov_b32 m0, %1\n\ts_waitcnt vmcnt(%4)\n\tbuffer_load_dword %2, %3, 0 offen lds\n\ts_mov_b32 m0, %0"
                         : "=&s"(keep) : "s"(lbase + L_XTRA), "v"(vo), "s"(xrs), "n"(DWIN - 1) : "memory");
          }
          STAMP(22);
        }
      }
      if (jt == 1) __builtin_amdgcn_s_barrier();
    }
  }
  u32 cl0 = 0;
  {
    wait_lgkm();
    if (WPF && !job) {
      if (HOLDW) __builtin_amdgcn_s_sleep(HOLDW);
      u32 keep;
      const u32 pv = (u32)(((slot * 16 + grp * 4 + panel - 4) * 8 + w) * 2048 + lane * 16);
      asm volatile("s_mov_b32 %0, m0\n\ts_mov_b32 m0, %1\n\ts_nop 0\n\tbuffer_load_dwordx4 %2, %3, 0 offen lds\n\t"
                   "buffer_load_dwordx4 %2, %3, 0 offen offset:1024 lds\n\ts_mov_b32 m0, %0\n\ts_waitcnt vmcnt(0)"
                   : "=&s"(keep) : "s"(lbase + L_XTRA + w * FRAG), "v"(pv), "s"(wrs) : "memory");
      STAMP(14);
      if (WPF > 1) __builtin_amdgcn_s_sleep(WPF);
    }
    dma_w(lbase + s01, wve, wvo, wrs, (u32)w * 65536u);
    dma_w(lbase + s01 + 8192, wve, wvo, wrs, (u32)(w + 8) * 65536u);
    if (!job) {
      if (w < 4) { u32 d, c, n; lut_entry(threadIdx.x, d, c, n); *(LDS u32x2*)(smem + L_LUT + threadIdx.x * 8) = u32x2{d, n}; }
      if (threadIdx.x < 8) *(LDS u32*)(smem + L_CTL + threadIdx.x * 4) = 0u;
      asm volatile("s_load_dword %0, %1, 0x0\n\ts_waitcnt lgkmcnt(0)" : "=&s"(call) : "s"(cntp) : "memory");
      call += 1u;
      __builtin_amdgcn_s_barrier();
    }
    STAMP(1);
    wait_vm<8>();
    STAMP(29);
    read_stage(0, s01);
    wait_lgkm();
    dma_w(lbase + s01, wve, wvo, wrs, (u32)(w + 16) * 65536u);
    wait_vm<8>();
    read_stage(1, s01 + 8192);
    wait_lgkm();
    dma_w(lbase + s01 + 8192, wve, wvo, wrs, (u32)(w + 24) * 65536u);
    STAMP(30);
    if (w == 1) cl0 = ld_agent(me + OFF_CLEAN + eo);
    wait_vm<8>();
    read_stage(2, s01);
    wait_vm<0>();
    read_stage(3, s01 + 8192);
  }
  call = __builtin_amdgcn_readfirstlane(call);
  const u32 par = call & 1u;
  STAMP(2);
  auto fail = [&]() { *(LDS u32*)(smem + L_CTL + 12) = 1u; *(LDS u32*)(smem + L_CTL + 20) = 1u; };
  if (w == 1) {
    u32 cl = __builtin_amdgcn_readfirstlane(cl0);
    if (cl < call) cl = lead();
    if (cl < call) cl = poll_all(me + OFF_CLEAN + eo, nullptr, call, tmo, call);
    if (lane == 0) {
      if (cl != call) fail();
      __hip_atomic_store((LDS u32*)(smem + L_CTL), 1u, __ATOMIC_RELAXED, __HIP_MEMORY_SCOPE_WORKGROUP);
      st_agent(me + OFF_CNT + b * LINE, call);
    }
  } else {
    while (lds_ld(smem + L_CTL) == 0u) __builtin_amdgcn_s_sleep(1);
  }
  asm volatile("" ::: "memory");
  STAMP(3);

  const auto rrs = __builtin_amdgcn_make_buffer_rsrc(p.ws[slot] + OFF_XC + par * XC_BYTES, 0, XC_BYTES, 0x00020000);
  const auto ors = __builtin_amdgcn_make_buffer_rsrc(p.out, 0, 4096 * NL * 4, 0x00020000);
  const u32 fi32 = (u32)(grp * 8 + w);
  const u32 vr = fi32 * FRAG + lane * 16;
  const float scale = *(const CST float*)p.sx * *(const CST float*)p.sw;
  auto put_partial = [&](const v4f* acc, int buf) {
#pragma unroll
    for (int c = 0; c < 4; ++c) *(LDS v4f*)(smem + s01 + buf * PBUF + r16 * PROW + c * 64 + gq * 16) = acc[c];
    u64 keep;
    asm volatile("s_mov_b64 %0, exec\n\ts_mov_b64 exec, 1\n\tds_add_u32 %1, %2\n\ts_mov_b64 exec, %0"
                 : "=&s"(keep) : "v"(lbase + L_CTL + 8 + buf * 8), "v"(1u) : "memory");
  };
  const u32 ov = ((u32)(grp * 16 + 2 * w + (lane >> 5)) * NL + panel * 64 + (lane & 31) * 2) * 4u | ((lane & 1) ? 0x80000000u : 0u);
  auto reduce = [&](int row0, int t) {
    const int buf = t & 1;
    LDS char* a = smem + L_STG + buf * PBUF + (2 * w + (lane >> 5)) * PROW + (lane & 31) * 8;
    v2f q[8];
    u32x2 ctl;
    while (true) {
      ctl = __builtin_bit_cast(u32x2, __hip_atomic_load((LDS u64*)(smem + L_CTL + 8 + buf * 8), __ATOMIC_RELAXED, __HIP_MEMORY_SCOPE_WORKGROUP));
      asm volatile("" ::: "memory");
#pragma unroll
      for (int i = 0; i < 8; ++i) q[i] = *(LDS v2f*)(a + i * L_STAGE);
      asm volatile("" : "+v"(q[0]), "+v"(q[1]), "+v"(q[2]), "+v"(q[3]), "+v"(q[4]), "+v"(q[5]), "+v"(q[6]), "+v"(q[7]));
      if (__builtin_amdgcn_readfirstlane(ctl.x) >= 8u * (u32)((t >> 1) + 1)) break;
      while (lds_ld(smem + L_CTL + 8 + buf * 8) < 8u * (u32)((t >> 1) + 1)) __builtin_amdgcn_s_sleep(1);
      asm volatile("" ::: "memory");
    }
    const v2f s = ((q[0] + q[1]) + (q[2] + q[3])) + ((q[4] + q[5]) + (q[6] + q[7]));
    const float sc_ = __builtin_amdgcn_readfirstlane(ctl.y) ? __builtin_nanf("") : scale;
    const v2f y = s * sc_;
    v2f d;
#pragma unroll
    for (int i = 0; i < 2; ++i) d[i] = __builtin_amdgcn_exp2f(__builtin_amdgcn_fmed3f(y[i], -60.f, 60.f) * -1.44269504f);
    d += 1.f;
    const v2f o = y * v2f{__builtin_amdgcn_rcpf(d[0]), __builtin_amdgcn_rcpf(d[1])};
    const v2i oi = __builtin_bit_cast(v2i, o);
    const int o0 = oi.x, o1 = oi.y;
    const u32x4 v = {(u32)o0, (u32)o1, (u32)__builtin_amdgcn_mov_dpp(o0, 0xb1, 0xf, 0xf, true), (u32)__builtin_amdgcn_mov_dpp(o1, 0xb1, 0xf, 0xf, true)};
    __builtin_amdgcn_raw_buffer_store_b128(v, ors, ov, (u32)row0 * (NL * 4), 17);
  };

  if (slot == 0) {
    const auto lrs = __builtin_amdgcn_make_buffer_rsrc(me, 0, WS_USED, 0x00020000);
    u32x4 nx[8];
    auto ldt = [&](int t) {
      const u32 so = t < 7 ? OFF_XC + par * XC_BYTES + XC_RAW + (u32)(t + 1) * (128 * FRAG) : OFF_XP + (par * NR + ((p.rank + t - 7) & 7)) * XP_BYTES;
#pragma unroll
      for (int i = 0; i < 8; ++i) nx[i] = __builtin_amdgcn_raw_buffer_load_b128(lrs, vr + (i & 1) * 1024, so + (i >> 1) * (32 * FRAG), 17);
    };
    auto row0 = [&](int t) { return t < 7 ? p.rank * 512 + (t + 1) * 64 : ((p.rank + t - 7) & 7) * 512; };
    ldt(0);
    for (int t = 0; t < 15; ++t) {
      u32x4 cu[8];
#pragma unroll
      for (int i = 0; i < 8; ++i) { cu[i] = nx[i]; asm volatile("" : "+v"(cu[i])); }
      if (t == 6) {
        const char* fl = me + OFF_PSH + gq * LINE + r16 * 4;
        if (poll_all(fl, fl + 4 * LINE, call, tmo, call) == ~0u && lane == 0) fail();
      }
      asm volatile("" ::: "memory");
      if (t < 14) ldt(t + 1);
      asm volatile("" ::: "memory");
      __builtin_amdgcn_sched_barrier(0);
      v4f acc[4] = {};
#pragma unroll
      for (int j = 0; j < 4; ++j) {
        const u32x4 a = cu[2 * j], c2 = cu[2 * j + 1];
        const v8i xv = {(int)a.x, (int)a.y, (int)a.z, (int)a.w, (int)c2.x, (int)c2.y, (int)c2.z, (int)c2.w};
#pragma unroll
        for (int c = 0; c < 4; ++c) acc[c] = __builtin_amdgcn_mfma_scale_f32_16x16x128_f8f6f4(Wf[j][c], xv, acc[c], 0, 0, 0, 0, 0, 0);
      }
      if (t > 0) reduce(row0(t - 1), t - 1);
      put_partial(acc, t & 1);
      if (t < 7) STAMP(5 + t);
    }
    reduce(row0(14), 14);
  } else {
    const u32 OOB = 0x80000000u;
    const u32 idx = __builtin_amdgcn_raw_buffer_load_b16(rrs, lane < 32 ? (u32)(lane * 4 + grp) * 16 + w * 2 : OOB, 0, 0);
    const u32 bs = (u32)lane * STEP + (u32)grp * BLK, len = (idx >> 9) & 63u;
    const u32 tl = len ? (bs + BLK_L + (idx & 511u) * 4) | ((len - 1u) << 24) : XC_BYTES;
    const u32 ta = len ? bs + w * 1024 : (idx & IDX_RAW) ? (XC_RAW + (u32)lane * (32 * FRAG) + fi32 * FRAG) | 0x80000000u : XC_BYTES;
    const u32 v16 = lane * 16, v8 = lane * 8, v4 = lane * 4, vt = lane < TOUCH ? lane * 64 : OOB;
    u32x4 A, B;
    u32x2 C;
    u32 L, pf = 0;
    u32 vl, sc, sl2, sa, sb, sp;
    auto plan = [&](int step) {
      const u32 e = __builtin_amdgcn_readlane(tl, step + 1), a = __builtin_amdgcn_readlane(ta, step);
      const u32 ma = (u32)((int)a >> 31);
      const u32 a1 = __builtin_amdgcn_readlane(ta, step + 1);
      sp = a1 & 0xbfffffffu;
      sc = sp + BLK_C - w * 512;
      sl2 = e & 0xffffffu;
      vl = min(v4, (e >> 22) & 0xfcu);
      sa = a & 0x7fffffffu;
      sb = ma ? sa : XC_BYTES;
      return ma != 0u;
    };
    auto ld_cl = [&]() {
      C = __builtin_amdgcn_raw_buffer_load_b64(rrs, v8, sc, 0);
      L = __builtin_amdgcn_raw_buffer_load_b32(rrs, vl, sl2, 0);
    };
    auto ld_ab = [&]() {
      A = __builtin_amdgcn_raw_buffer_load_b128(rrs, v16, sa, 0);
      B = __builtin_amdgcn_raw_buffer_load_b128(rrs, v16 + 1024, sb, 0);
      if (TOUCH) {
        asm volatile("" :: "v"(pf));
        pf = __builtin_amdgcn_raw_buffer_load_b32(rrs, vt, sp, 0);
      }
    };
    plan(3);
    ld_cl();
    u32x2 c1 = C;
    u32 l1 = L;
    asm volatile("" ::: "memory");
    bool raw1 = plan(4);
    ld_cl();
    ld_ab();
    asm volatile("" ::: "memory");
    __builtin_amdgcn_sched_barrier(0);
    v8i ex = exponents(c1, l1, smem + L_LUT);
    for (int r = 1; r < 8; ++r) {
      v4f acc[4] = {};
#pragma unroll
      for (int j = 0; j < 4; ++j) {
        const int step = r * 4 + j;
        const bool raw = raw1;
        raw1 = plan(step + 1);
        asm volatile("" : "+v"(ex), "+v"(vl), "+s"(sc), "+s"(sl2), "+s"(sa), "+s"(sb), "+s"(sp));
        __builtin_amdgcn_sched_barrier(0);
        u32x4 a = A, b2 = B;
        c1 = C; l1 = L;
        asm volatile("" : "+v"(a), "+v"(b2), "+v"(c1), "+v"(l1));
        if (r == SR) STAMP(25 + j);
        ld_cl();
        ld_ab();
        asm volatile("" ::: "memory");
        __builtin_amdgcn_sched_barrier(0);
        if (__builtin_expect(raw, 0)) {
#pragma unroll
          for (int i = 0; i < 8; ++i) asm volatile("v_mov_b32 %0, %1" : "=v"(ex[i]) : "v"(i < 4 ? a[i] : b2[i - 4]));
        } else {
          ex = merge(a, ex);
        }
#pragma unroll
        for (int c = 0; c < 4; ++c) acc[c] = __builtin_amdgcn_mfma_scale_f32_16x16x128_f8f6f4(Wf[j][c], ex, acc[c], 0, 0, 0, 0, 0, 0);
        if (j == 1 && r > 1) reduce(sh * 512 + (r - 1) * 64, r - 2);
        if (j == 3) put_partial(acc, (r - 1) & 1);
        if (step < 31) ex = exponents(c1, l1, smem + L_LUT);
      }
      STAMP(4 + r);
    }
    reduce(sh * 512 + 7 * 64, 6);
  }
  STAMP(12);
  STAMP(13);
  STAMP_FLUSH();
}

extern "C" void kernel_launch(int rank, int nranks, ncclComm_t comm,
                              hipStream_t stream,
                              void* const* d_in, const int* in_sizes, int n_in,
                              void* d_out, int out_size,
                              const int* in_dtypes, int out_dtype,
                              void* const* d_out_all,
                              void* const* d_ws_all, size_t ws_bytes) {
  if (nranks != NR || (unsigned)rank >= (unsigned)NR || n_in != 4 || ws_bytes < (size_t)WS_USED + (1u << 20) ||
      in_sizes[0] != 512 * KD || in_sizes[1] != KD * NL || out_size != 4096 * NL) {
    fprintf(stderr, "unexpected problem shape\n");
    return;
  }
  Params p;
  for (int i = 0; i < NR; ++i) p.ws[i] = (char*)d_ws_all[(rank + i) & 7];
  p.x = (const unsigned char*)d_in[0];
  p.w = (const unsigned char*)d_in[1];
  p.sx = (const float*)d_in[2];
  p.sw = (const float*)d_in[3];
  p.out = (float*)d_out;
  p.rank = rank;
  p.pad = 0;
  ag_gemm<<<256, 512, 0, stream>>>(p);
  HIP_CHECK(hipGetLastError());
}
```

```cpp
#include <hip/hip_runtime.h>
#include <rccl/rccl.h>
#include <cstdio>

#define HIP_CHECK(expr) do { hipError_t _e = (expr); if (_e != hipSuccess) { \
    fprintf(stderr, "HIP error %s at %s:%d\n", hipGetErrorString(_e), __FILE__, __LINE__); \
    return; } } while (0)

typedef unsigned u32;
typedef unsigned long long u64;
typedef u32 u32x4 __attribute__((ext_vector_type(4)));
typedef u32 u32x2 __attribute__((ext_vector_type(2)));
typedef int v2i __attribute__((ext_vector_type(2)));
typedef int v4i __attribute__((ext_vector_type(4)));
typedef int v8i __attribute__((ext_vector_type(8)));
typedef float v4f __attribute__((ext_vector_type(4)));
typedef float v2f __attribute__((ext_vector_type(2)));
#define LDS __attribute__((address_space(3)))
#define GLB __attribute__((address_space(1)))
#define CST __attribute__((address_space(4)))

constexpr int NR = 8;
constexpr int KD = 4096, NL = 512;
constexpr u32 FRAG = 2048;
constexpr u32 LINE = 128;
constexpr u32 OFF_CNT = 0;
constexpr u32 OFF_TMO = OFF_CNT + 256 * LINE;
constexpr u32 OFF_ELECT = OFF_TMO + LINE;
constexpr u32 OFF_CLEAN = OFF_ELECT + 64 * LINE;
constexpr u32 OFF_CPY = OFF_CLEAN + 64 * LINE;
constexpr u32 CPY_GO = 112 * 8;
constexpr u32 OFF_PSH = OFF_CPY + 2 * NR * 1024;
constexpr u32 L2_DW = 63;
constexpr u32 BLK_C = 8 * 1024, BLK_L = 8 * 1536;
constexpr u32 BLK = BLK_L + 2048;
constexpr u32 STEP = 4 * BLK;
constexpr u32 OFF_XC = 1u << 20;
constexpr u32 XC_RAW = 2u << 20, XC_BYTES = XC_RAW + 1024 * FRAG;
constexpr u32 OFF_XP = OFF_XC + 2 * XC_BYTES;
constexpr u32 XP_BYTES = 128 * FRAG;
constexpr u32 WS_USED = OFF_XP + 2 * NR * XP_BYTES;
static_assert(32 * STEP <= XC_RAW && STEP % 128 == 0, "layout");
#define STAMP(i) do {} while (0)
#define STAMP_INIT() do {} while (0)
#define STAMP_FLUSH() do {} while (0)
#define L_STAMP 0
constexpr u32 L_LUT = 0;
constexpr u32 L_STG = 2048, L_STAGE = 16384;
constexpr u32 L_XTRA = L_STG + 8 * L_STAGE;
constexpr u32 L_CTL = L_XTRA + 8 * FRAG;
constexpr u32 L_BYTES = L_CTL + 64 + L_STAMP;
constexpr u32 PROW = 272;
constexpr u32 PBUF = 16 * PROW;
#define PW 3
#define GOS 9
#define TOUCH 2
#define ROT 1
#define SELFD 100
#define D1 10
#define ENDRAW 1
#define HOLDW 24
#define SR 6
#define WPF 1
#define DWIN 16
constexpr u64 DEADLINE_TICKS = 50000000ull;

struct Params {
  char* ws[NR];
  const unsigned char* x;
  const unsigned char* w;
  const float* sx;
  const float* sw;
  float* out;
  int rank;
  int pad;
};

template <int N> __device__ __forceinline__ void wait_vm() { asm volatile("s_waitcnt vmcnt(%0)" :: "n"(N) : "memory"); }
__device__ __forceinline__ void wait_lgkm() { asm volatile("s_waitcnt lgkmcnt(0)" ::: "memory"); }
__device__ __forceinline__ u32 ld_sys(const char* a) { return __hip_atomic_load((GLB u32*)a, __ATOMIC_RELAXED, __HIP_MEMORY_SCOPE_SYSTEM); }
__device__ __forceinline__ u32 ld_agent(const char* a) { return __hip_atomic_load((GLB u32*)a, __ATOMIC_RELAXED, __HIP_MEMORY_SCOPE_AGENT); }
__device__ __forceinline__ void st_agent(char* a, u32 v) { __hip_atomic_store((GLB u32*)a, v, __ATOMIC_RELAXED, __HIP_MEMORY_SCOPE_AGENT); }
__device__ __forceinline__ u32 lds_ld(LDS char* a) { return __hip_atomic_load((LDS u32*)a, __ATOMIC_RELAXED, __HIP_MEMORY_SCOPE_WORKGROUP); }

__device__ __forceinline__ void dma_w(u32 lds, u32 ve, u32 vo, u32x4 rs, u32 so) {
  u32 keep;
#define DMA_STEP(V) "s_add_u32 m0, m0, 0x400\n\ts_add_u32 %1, %1, 0x2000\n\tbuffer_load_dwordx4 " V ", %5, %1 offen lds\n\t"
  asm volatile("s_mov_b32 %0, m0\n\ts_mov_b32 m0, %2\n\ts_nop 0\n\tbuffer_load_dwordx4 %3, %5, %1 offen lds\n\t"
               DMA_STEP("%4") DMA_STEP("%3") DMA_STEP("%4") DMA_STEP("%3") DMA_STEP("%4") DMA_STEP("%3") DMA_STEP("%4")
               "s_mov_b32 m0, %0"
               : "=&s"(keep), "+s"(so) : "s"(lds), "v"(ve), "v"(vo), "s"(rs) : "memory", "scc");
#undef DMA_STEP
}
__device__ __forceinline__ void st16_sys(char* base, u32 off, u32x4 v) {
  asm volatile("global_store_dwordx4 %0, %1, %2 sc0 sc1\n\ts_nop 1" :: "v"(off), "v"(v), "s"(base) : "memory");
}
__device__ __forceinline__ void st8_sys(char* base, u32 off, u32x2 v) {
  asm volatile("global_store_dwordx2 %0, %1, %2 sc0 sc1\n\ts_nop 1" :: "v"(off), "v"(v), "s"(base) : "memory");
}
__device__ __forceinline__ void st4_sys(char* base, u32 off, u32 v) {
  asm volatile("global_store_dword %0, %1, %2 sc0 sc1\n\ts_nop 1" :: "v"(off), "v"(v), "s"(base) : "memory");
}
__device__ __forceinline__ u32 poll_all(const char* word, const char* word2, u32 target, char* tmo, u32 call) {
  const u64 t0 = __builtin_amdgcn_s_memrealtime();
  for (u32 it = 1;; ++it) {
    const u32 v = ld_sys(word), v2 = word2 ? ld_sys(word2) : v;
    if (__all(v >= target && v2 >= target)) return __builtin_amdgcn_readfirstlane(v);
    if ((it & 15u) == 0u && (ld_agent(tmo) == call || __builtin_amdgcn_s_memrealtime() - t0 > DEADLINE_TICKS)) { st_agent(tmo, call); return ~0u; }
    __builtin_amdgcn_s_sleep(1);
  }
}

__device__ __forceinline__ u64 poll_tag(const char* word, const char* word2, char* tmo, u32 call) {
  const u64 t0 = __builtin_amdgcn_s_memrealtime();
  for (u32 it = 1;; ++it) {
    const u64 v = __hip_atomic_load((GLB u64*)word, __ATOMIC_RELAXED, __HIP_MEMORY_SCOPE_SYSTEM);
    const u64 v2 = word2 ? __hip_atomic_load((GLB u64*)word2, __ATOMIC_RELAXED, __HIP_MEMORY_SCOPE_SYSTEM) : v;
    if (__all((u32)(v >> 48) == (call & 0xffffu) && (u32)(v2 >> 48) == (call & 0xffffu))) return v;
    if ((it & 15u) == 0u && (ld_agent(tmo) == call || __builtin_amdgcn_s_memrealtime() - t0 > DEADLINE_TICKS)) { st_agent(tmo, call); return (u64)(~call) << 48; }
    __builtin_amdgcn_s_sleep(1);
  }
}

constexpr u32 L2_CAP = (L2_DW * 32 - 8) / 3;

__device__ __forceinline__ u32 wave_scan(u32 v) {
  v += __builtin_amdgcn_update_dpp(0u, v, 0x111, 0xf, 0xf, false);
  v += __builtin_amdgcn_update_dpp(0u, v, 0x112, 0xf, 0xf, false);
  v += __builtin_amdgcn_update_dpp(0u, v, 0x114, 0xf, 0xf, false);
  v += __builtin_amdgcn_update_dpp(0u, v, 0x118, 0xf, 0xf, false);
  v += __builtin_amdgcn_update_dpp(0u, v, 0x142, 0xa, 0xf, false);
  v += __builtin_amdgcn_update_dpp(0u, v, 0x143, 0xc, 0xf, false);
  return v;
}
__device__ __forceinline__ void lut_entry(u32 cb, u32& dsel, u32& csel, u32& s3) {
  u32 n = 0; dsel = 0; csel = 0x0c0c0c0cu;
#pragma unroll
  for (int b = 0; b < 4; ++b) {
    const u32 c = (cb >> (2 * b)) & 3u;
    const bool e = c == 3u;
    dsel |= (e ? 4u + n : c) << (8 * b);
    csel = e ? (csel & ~(0xffu << (8 * n))) | ((u32)b << (8 * n)) : csel;
    n += e;
  }
  s3 = 3u * n;
}
struct Packed { u32x4 n; u32x2 c; u32 l; };

template <int B> __device__ __forceinline__ u32 byte_shl(u32 c, u32 sh) {
  u32 r;
  if (B == 0) asm("v_lshlrev_b32_sdwa %0, %1, %2 dst_sel:DWORD dst_unused:UNUSED_PAD src0_sel:DWORD src1_sel:BYTE_0" : "=v"(r) : "v"(sh), "v"(c));
  if (B == 1) asm("v_lshlrev_b32_sdwa %0, %1, %2 dst_sel:DWORD dst_unused:UNUSED_PAD src0_sel:DWORD src1_sel:BYTE_1" : "=v"(r) : "v"(sh), "v"(c));
  if (B == 2) asm("v_lshlrev_b32_sdwa %0, %1, %2 dst_sel:DWORD dst_unused:UNUSED_PAD src0_sel:DWORD src1_sel:BYTE_2" : "=v"(r) : "v"(sh), "v"(c));
  if (B == 3) asm("v_lshlrev_b32_sdwa %0, %1, %2 dst_sel:DWORD dst_unused:UNUSED_PAD src0_sel:DWORD src1_sel:BYTE_3" : "=v"(r) : "v"(sh), "v"(c));
  return r;
}
__device__ __forceinline__ u32 encode(const v8i& xi, Packed& o, LDS char* lutc, LDS char* buf, int lane) {
  u32 x[8];
#pragma unroll
  for (int i = 0; i < 8; ++i) x[i] = (u32)xi[i];
  u32 se = 0;
#pragma unroll
  for (int i = 0; i < 8; i += 2) se = __builtin_amdgcn_sad_u8(x[i] & 0x78787878u, 0u, se);
  const u32 tot = __builtin_amdgcn_readlane(wave_scan(se), 63);
  const u32 e0 = min((tot < 256u ? 0u : tot - 256u) >> 13, 10u);
  const u32 bias = (13u - e0) * 0x01010101u;
  u32 inv1 = 0, inv2 = 0, ca[8], kb[8];
#pragma unroll
  for (int i = 0; i < 8; ++i) {
    const u32 gb = ((x[i] >> 3) & 0x0f0f0f0fu) + bias;
    const u32 sel = gb & 0x07070707u;
    const u32 a = __builtin_amdgcn_perm(0x1008001cu, 0x1b1a1918u, sel);
    const u32 b = __builtin_amdgcn_perm(0x98989898u, 0x981f1e1du, sel);
    const u32 m = __builtin_amdgcn_perm(0u, 0u, __builtin_amdgcn_bitop3_b32(gb >> 4, 0x01010101u, 0x0c0c0c0cu, 0xea));
    const u32 code = __builtin_amdgcn_bitop3_b32(m, b, a, 0xca);
    inv1 = __builtin_amdgcn_bitop3_b32(inv1, gb, gb >> 1, 0xf9);
    inv2 |= code;
    ca[i] = __builtin_amdgcn_udot4(code & 0x18181818u, 0x40100401u, 0u, false);
    kb[i] = code & 0x07070707u;
  }
  u64 acc = 0; u32 n3 = 0;
#pragma unroll
  for (int i = 7; i >= 0; --i) {
    const u32x2 t = *(LDS u32x2*)(lutc + ca[i]);
    const u32 comp = __builtin_amdgcn_perm(kb[i], kb[i], t.x);
    acc = (acc << t.y) | __builtin_amdgcn_udot4(comp, 0x00400801u, byte_shl<3>(comp, 9u), false);
    n3 += t.y;
  }
  const u32 incl = wave_scan(n3);
  const u32 all3 = __builtin_amdgcn_readlane(incl, 63);
  const bool fit = !__any(((inv1 & 0x08080808u) | (inv2 & 0x80808080u)) != 0u || n3 > 63u) && all3 <= 3u * L2_CAP;
  o.c = u32x2{(ca[0] >> 3) | (ca[1] << 5) | (ca[2] << 13) | (ca[3] << 21), (ca[4] >> 3) | (ca[5] << 5) | (ca[6] << 13) | (ca[7] << 21)};
#pragma unroll
  for (int j = 0; j < 4; ++j) {
    const u32 od = x[2 * j + 1] & 0x87878787u;
    o.n[j] = (x[2 * j] & 0x87878787u) | __builtin_amdgcn_alignbit(od, od, 28);
  }
  o.l = 0;
  if (!fit) return 0u;
  const u32 off = incl - n3 + 8u, sh = off & 31u;
  const u64 lo = acc << sh;
  const u32 hi = ((u32)(acc >> 32) >> 1) >> (31u - sh);
  LDS u32* d = (LDS u32*)(buf + ((off >> 5) << 2));
  __hip_atomic_fetch_or(d, (u32)lo | (lane == 0 ? e0 : 0u), __ATOMIC_RELAXED, __HIP_MEMORY_SCOPE_WAVEFRONT);
  __hip_atomic_fetch_or(d + 1, (u32)(lo >> 32), __ATOMIC_RELAXED, __HIP_MEMORY_SCOPE_WAVEFRONT);
  __hip_atomic_fetch_or(d + 2, hi, __ATOMIC_RELAXED, __HIP_MEMORY_SCOPE_WAVEFRONT);
  o.l = __hip_atomic_load((LDS u32*)(buf + lane * 4), __ATOMIC_RELAXED, __HIP_MEMORY_SCOPE_WAVEFRONT);
  return (all3 + 8u + 31u) >> 5;
}

__device__ __forceinline__ v8i exponents(u32x2 c, u32 l, LDS char* lutd) {
  u32x2 t[8];
  t[0] = *(LDS u32x2*)(lutd + byte_shl<0>(c.x, 3u)); t[1] = *(LDS u32x2*)(lutd + byte_shl<1>(c.x, 3u));
  t[2] = *(LDS u32x2*)(lutd + byte_shl<2>(c.x, 3u)); t[3] = *(LDS u32x2*)(lutd + byte_shl<3>(c.x, 3u));
  t[4] = *(LDS u32x2*)(lutd + byte_shl<0>(c.y, 3u)); t[5] = *(LDS u32x2*)(lutd + byte_shl<1>(c.y, 3u));
  t[6] = *(LDS u32x2*)(lutd + byte_shl<2>(c.y, 3u)); t[7] = *(LDS u32x2*)(lutd + byte_shl<3>(c.y, 3u));
  __builtin_amdgcn_sched_barrier(0);
  const u32 n = __builtin_popcount(__builtin_amdgcn_bitop3_b32(c.x, c.x >> 1, 0x55555555u, 0x80)) +
                __builtin_popcount(__builtin_amdgcn_bitop3_b32(c.y, c.y >> 1, 0x55555555u, 0x80));
  u32 incl = wave_scan(n);
  asm("" : "+v"(incl));
  const u32 off = __umul24(incl - n, 3u) + 8u;
  const int a = (int)(off >> 3);
  const u32 w0 = __builtin_amdgcn_ds_bpermute(a, l), w1 = __builtin_amdgcn_ds_bpermute(a + 4, l), w2 = __builtin_amdgcn_ds_bpermute(a + 8, l);
  u64 win = ((u64)__builtin_amdgcn_alignbit(w2, w1, off) << 32) | __builtin_amdgcn_alignbit(w1, w0, off);
  const u32 e8 = (__builtin_amdgcn_readfirstlane(l) & 15u) * 0x08080808u;
  const u32 td = 0x00100800u + e8, tl = (0x70686058u + e8) & 0x7f7f7f7fu, th = (0xa8a09878u + e8) & 0x7f7f7f7fu;
  v8i o;
#pragma unroll
  for (int i = 0; i < 8; ++i) {
    const u32 a0 = (u32)win;
    const u32 y = __builtin_amdgcn_perm(a0 >> 6, a0, 0x0c040c00u);
    const u32 z = __builtin_amdgcn_bitop3_b32(y, y << 5, 0x07070707u, 0xa8);
    o[i] = (int)__builtin_amdgcn_perm(__builtin_amdgcn_perm(th, tl, z), td, t[i].x);
    if (i < 7) win >>= t[i].y;
  }
  return o;
}
__device__ __forceinline__ v8i merge(u32x4 n, v8i ex) {
#pragma unroll
  for (int i = 0; i < 8; ++i) {
    const u32 nb = (i & 1) ? __builtin_amdgcn_alignbit(n[i >> 1], n[i >> 1], 4) : n[i >> 1];
    ex[i] = (int)__builtin_amdgcn_bitop3_b32(nb, 0x87878787u, (u32)ex[i], 0xea);
  }
  return ex;
}

template <int WIN> __device__ __forceinline__ void drive(const char* xc, u32 lds, int lane, int n) {
  const u64 xa = (u64)xc;
  const u32x4 xrs = {(u32)xa, (u32)(xa >> 32) & 0xffffu, XC_RAW, 0x00020000u};
  u32 keep;
  for (int i = 0; i < n * 4; ++i) {
    const u32 bi = (u32)(i >> 2), li = (i & 3) * 27 + lane;
    const u32 vo = lane < 27 && li < 107 ? 4 * STEP + bi * BLK + li * 128 : 0x80000000u;
    asm volatile("s_mov_b32 %0, m0\n\ts_mov_b32 m0, %1\n\ts_waitcnt vmcnt(%4)\n\tbuffer_load_dword %2, %3, 0 offen lds\n\ts_mov_b32 m0, %0"
                 : "=&s"(keep) : "s"(lds), "v"(vo), "s"(xrs), "n"(WIN - 1) : "memory");
  }
}

__global__ void __launch_bounds__(512, 2) ag_gemm(Params p) {
  __shared__ __attribute__((aligned(16))) char smem_[L_BYTES];
  LDS char* const smem = (LDS char*)smem_;
  const int lane = threadIdx.x & 63;
  const int w = __builtin_amdgcn_readfirstlane(threadIdx.x >> 6);
  const int b = blockIdx.x, slot = b & 7, q = b >> 3, grp = q >> 3, panel = q & 7;
  const int gq = lane >> 4, r16 = lane & 15;
  const unsigned char* px;
  char *me, *dst;
  const int pc = slot * 2 + (q >> 4);
  const int di = ROT ? (w == 7 ? 0 : (w + pc) % 7 + 1) : (w + 1) & 7;
  asm volatile("s_load_dwordx2 %0, %3, 0x40\n\ts_load_dwordx2 %1, %3, 0x0\n\ts_load_dwordx2 %2, %3, %4"
               : "=&s"(px), "=&s"(me), "=&s"(dst) : "s"(__builtin_amdgcn_kernarg_segment_ptr()), "s"(di * 8));
  const bool job = panel < 4, pushcu = (q & 15) == 0;
  const int jt = grp * 4 + panel;
  const bool coord = jt == 1;
  const int Fp = w * 16 + ((pc + 2 * w) & 15), Fc = 128 + (slot * 14 + jt - 1 - (jt > 8)) * 8 + w;
  const int F = Fc + ((Fp - Fc) & -(int)pushcu);
  u32 xo = (u32)((F >> 7) * 64 + ((F >> 3) & 3) * 16 + r16) * KD + ((((F >> 5) & 3) * 8 + (F & 7)) * 128 + gq * 16);
  asm volatile("s_waitcnt lgkmcnt(0)" : "+v"(xo), "+s"(px), "+s"(me), "+s"(dst));
  const char* const cntp = me + OFF_CNT + b * LINE;
  u32 call;
  u32x4 x0, x1;
  if (job)
    asm volatile("global_load_dwordx4 %0, %3, %5\n\tglobal_load_dwordx4 %1, %3, %5 offset:64\n\ts_load_dword %2, %4, 0x0"
                 : "=&v"(x0), "=&v"(x1), "=&s"(call) : "v"(xo), "s"(cntp), "s"(px) : "memory");
  STAMP_INIT();
  const u64 wa = (u64)p.w;
  const u32x4 wrs = {(u32)wa, (u32)(wa >> 32) & 0xffffu, (u32)(KD * NL), 0x00020000u};
  const u32 wve = (u32)(lane >> 2) * NL + panel * 64 + (((lane & 3) ^ (gq & 1)) << 4), wvo = wve ^ 32u;
  const u32 lbase = (u32)(size_t)smem;
  const u32 s01 = L_STG + w * L_STAGE;
  v8i Wf[4][4];
  const int q8 = r16 >> 1, f2 = ((q8 >> 2) & 1) | ((gq & 1) << 1);
  const u32 tra = (u32)(gq * 16 + q8) * 64 + (r16 & 1) * 8;
  auto read_stage = [&](int j, u32 off) {
#pragma unroll
    for (int c = 0; c < 4; ++c) {
      LDS char* a = smem + off + tra + ((c ^ f2) << 4);
      v2i t0 = __builtin_amdgcn_ds_read_tr8_b64_v2i32((LDS v2i*)(a));
      v2i t1 = __builtin_amdgcn_ds_read_tr8_b64_v2i32((LDS v2i*)(a + 512));
      v2i t2 = __builtin_amdgcn_ds_read_tr8_b64_v2i32((LDS v2i*)(a + 4096));
      v2i t3 = __builtin_amdgcn_ds_read_tr8_b64_v2i32((LDS v2i*)(a + 4608));
      Wf[j][c] = v8i{t0.x, t0.y, t1.x, t1.y, t2.x, t2.y, t3.x, t3.y};
    }
  };
  const int sh = (p.rank + slot) & 7;
  char* const tmo = me + OFF_TMO;
  u32 xcc;
  asm volatile("s_getreg_b32 %0, hwreg(HW_REG_XCC_ID)" : "=s"(xcc));
  const u32 eo = ((xcc & 7u) * 8 + slot) * LINE;
  auto lead = [&]() {
    u32 old = 0;
    if (lane == 0) old = __hip_atomic_fetch_max((GLB u32*)(me + OFF_ELECT + eo), call, __ATOMIC_RELAXED, __HIP_MEMORY_SCOPE_AGENT);
    if (__builtin_amdgcn_readfirstlane(old) >= call) return 0u;
    const char* fl = me + OFF_CPY + ((call & 1u) * NR + sh) * 1024;
    const u32 cl = __builtin_amdgcn_readfirstlane((u32)(poll_tag(fl + lane * 8, fl + 512 + lane * 8, tmo, call) >> 48)) == (call & 0xffffu) ? call : ~0u;
    STAMP(20);
    __builtin_amdgcn_fence(__ATOMIC_ACQUIRE, "");
    wait_vm<0>();
    STAMP(21);
    if (lane == 0) st_agent(me + OFF_CLEAN + eo, cl);
    return cl;
  };
  auto tables = [&]() {
    if (w < 4) { u32 d, c, n; lut_entry(threadIdx.x, d, c, n); *(LDS u32x2*)(smem + L_LUT + threadIdx.x * 8) = u32x2{d, n}; }
    if (threadIdx.x < 8) *(LDS u32*)(smem + L_CTL + threadIdx.x * 4) = 0u;
  };
  if (job) {
    if (!pushcu) {
#pragma unroll
      for (int i = 0; i < 4; ++i) {
        u32 d, c, n;
        lut_entry(lane + 64 * i, d, c, n);
        *(LDS u32x2*)(smem + s01 + (lane + 64 * i) * 8) = u32x2{c, n};
      }
      *(LDS u32*)(smem + s01 + 2048 + lane * 4) = 0u;
    }
    asm volatile("s_waitcnt vmcnt(0) lgkmcnt(0)" : "+v"(x0), "+v"(x1), "+s"(call) :: "memory");
    call += 1u;
    STAMP(14);
    LDS char* sl = smem + L_XTRA + w * FRAG;
    if (__builtin_expect(pushcu, 1)) {
      const u32 off = OFF_XP + ((call & 1u) * NR + p.rank) * XP_BYTES + lane * 16;
      st16_sys(dst, off + F * FRAG, x0); st16_sys(dst, off + F * FRAG + 1024, x1);
      *(LDS u32x4*)(sl + lane * 16) = x0;
      *(LDS u32x4*)(sl + 1024 + lane * 16) = x1;
      wait_lgkm();
      __builtin_amdgcn_s_barrier();
      asm volatile("" ::: "memory");
      if (SELFD && w == 7) __builtin_amdgcn_s_sleep(SELFD);
#pragma unroll
      for (int i = 1; i < 8; ++i) {
        const int fi = (i + w) & 7, Fi = fi * 16 + ((pc + 2 * fi) & 15);
        const u32x4 a = *(LDS u32x4*)(smem + L_XTRA + fi * FRAG + lane * 16), c = *(LDS u32x4*)(smem + L_XTRA + fi * FRAG + 1024 + lane * 16);
#pragma unroll
        for (int h = 0; h < 2; ++h) {
          const int k = 2 * i + h;
          if (k >= PW) { if (k > GOS && k < GOS + PW) wait_vm<PW>(); else wait_vm<PW - 1>(); }
          if (k == 6) STAMP(24);
          if (k == 10) STAMP(20);
          if (k == 14) STAMP(21);
          if (k == GOS && lane == 0) st8_sys(dst, OFF_CPY + ((call & 1u) * NR + p.rank) * 1024 + CPY_GO + pc * 8, u32x2{0u, call << 16});
          st16_sys(dst, off + Fi * FRAG + h * 1024, h ? c : a);
        }
      }
      wait_vm<0>();
      STAMP(19);
      if (lane == 0) st4_sys(dst, OFF_PSH + p.rank * LINE + pc * 4, call);
      tables();
      wait_lgkm();
      __builtin_amdgcn_s_barrier();
    } else {
      const u32 base = OFF_XC + (call & 1u) * XC_BYTES;
      st16_sys(me, base + XC_RAW + F * FRAG + lane * 16, x0); st16_sys(me, base + XC_RAW + F * FRAG + 1024 + lane * 16, x1);
      Packed pk;
      const u32 len = F >= 1024 - 32 * ENDRAW ? 0u : encode(v8i{(int)x0.x, (int)x0.y, (int)x0.z, (int)x0.w, (int)x1.x, (int)x1.y, (int)x1.z, (int)x1.w}, pk, smem + s01, smem + s01 + 2048, lane);
      STAMP(24);
      const u32 blk = base + (u32)(F >> 3) * BLK;
      if (lane == 0) *(LDS u32*)(smem + L_CTL + 32 + w * 4) = len;
      if (len) {
        *(LDS u32x2*)(sl + lane * 8) = pk.c;
        st16_sys(me, blk + w * 1024 + lane * 16, pk.n);
        const u32x4 c = *(LDS u32x4*)(sl + lane * 16);
        if (lane < 32) st16_sys(me, blk + BLK_C + w * 512 + lane * 16, c);
      }
      wait_lgkm();
      __builtin_amdgcn_s_barrier();
      asm volatile("" ::: "memory");
      const u32 ln = lane < 8 ? *(LDS u32*)(smem + L_CTL + 32 + lane * 4) : 0u, st = wave_scan(ln) - ln;
      if (lane < (int)len) st4_sys(me, blk + BLK_L + ((u32)__builtin_amdgcn_readlane(st, w) + lane) * 4, pk.l);
      u64 fo = (u64)(call & 0xffffu) << 48;
#pragma unroll
      for (int i = 0; i < 8; ++i) fo |= (u64)(u32)__builtin_amdgcn_readlane(ln, i) << (6 * i);
      tables();
      char* wsl = p.ws[0];
#pragma unroll
      for (int i = 1; i < NR; ++i) wsl = (lane & 7) == i ? p.ws[i] : wsl;
      wsl += OFF_CPY + ((call & 1u) * NR + p.rank) * 1024 + ((F - 128) >> 3) * 8;
      asm volatile("s_waitcnt vmcnt(0) lgkmcnt(0)" : "+v"(wsl) :: "memory");
      __builtin_amdgcn_s_barrier();
      STAMP(19);
      if (w == 0) {
        if (lane < NR) asm volatile("global_store_dwordx2 %0, %1, off sc0 sc1" :: "v"(wsl), "v"(fo) : "memory");
        if (coord && lead() != 0u && slot != 0) {
          STAMP(23);
          drive<DWIN>(p.ws[slot] + base, lbase + L_XTRA, lane, D1);
          STAMP(22);
        }
      }
      if (coord) __builtin_amdgcn_s_barrier();
    }
  }
  u32 cl0 = 0;
  u64 fv;
  const char* lenp;
  {
    wait_lgkm();
    if (WPF && !job) {
      if (HOLDW) __builtin_amdgcn_s_sleep(HOLDW);
      u32 keep;
      const u32 pv = (u32)(((slot * 16 + grp * 4 + panel - 4) * 8 + w) * 2048 + lane * 16);
      asm volatile("s_mov_b32 %0, m0\n\ts_mov_b32 m0, %1\n\ts_nop 0\n\tbuffer_load_dwordx4 %2, %3, 0 offen lds\n\t"
                   "buffer_load_dwordx4 %2, %3, 0 offen offset:1024 lds\n\ts_mov_b32 m0, %0\n\ts_waitcnt vmcnt(0)"
                   : "=&s"(keep) : "s"(lbase + L_XTRA + w * FRAG), "v"(pv), "s"(wrs) : "memory");
      STAMP(14);
      if (WPF > 1) __builtin_amdgcn_s_sleep(WPF);
    }
    dma_w(lbase + s01, wve, wvo, wrs, (u32)w * 65536u);
    dma_w(lbase + s01 + 8192, wve, wvo, wrs, (u32)(w + 8) * 65536u);
    if (!job) {
      tables();
      asm volatile("s_load_dword %0, %1, 0x0\n\ts_waitcnt lgkmcnt(0)" : "=&s"(call) : "s"(cntp) : "memory");
      call += 1u;
      __builtin_amdgcn_s_barrier();
    }
    STAMP(1);
    wait_vm<8>();
    STAMP(29);
    read_stage(0, s01);
    wait_lgkm();
    dma_w(lbase + s01, wve, wvo, wrs, (u32)(w + 16) * 65536u);
    wait_vm<8>();
    read_stage(1, s01 + 8192);
    wait_lgkm();
    dma_w(lbase + s01 + 8192, wve, wvo, wrs, (u32)(w + 24) * 65536u);
    STAMP(30);
    lenp = me + OFF_CPY + ((call & 1u) * NR + sh) * 1024 + ((min(max(lane, 4), 31) - 4) * 4 + grp) * 8;
    fv = __hip_atomic_load((GLB u64*)lenp, __ATOMIC_RELAXED, __HIP_MEMORY_SCOPE_SYSTEM);
    cl0 = ld_agent(me + OFF_CLEAN + eo);
    wait_vm<8>();
    read_stage(2, s01);
    wait_vm<0>();
    read_stage(3, s01 + 8192);
  }
  call = __builtin_amdgcn_readfirstlane(call);
  const u32 par = call & 1u;
  STAMP(2);
  auto fail = [&]() { *(LDS u32*)(smem + L_CTL + 12) = 1u; *(LDS u32*)(smem + L_CTL + 20) = 1u; };
  u32 len = 0, lst = 0;
  if (slot != 0) {
    if (!__all((u32)(fv >> 48) == (call & 0xffffu))) {
      fv = poll_tag(lenp, nullptr, tmo, call);
      if (!__all((u32)(fv >> 48) == (call & 0xffffu)) && lane == 0) fail();
    }
#pragma unroll
    for (int i = 0; i < 7; ++i) lst += i < w ? (u32)(fv >> (6 * i)) & 63u : 0u;
    len = (u32)(fv >> (6 * w)) & 63u;
    asm volatile("" : "+v"(len), "+v"(lst));
  }
  {
    u32 cl = __builtin_amdgcn_readfirstlane(cl0);
    for (int i = 0; i < 16 && cl < call; ++i) cl = __builtin_amdgcn_readfirstlane(ld_agent(me + OFF_CLEAN + eo));
    if (cl < call && w == 1) cl = lead();
    if (cl < call) cl = poll_all(me + OFF_CLEAN + eo, nullptr, call, tmo, call);
    if (lane == 0) {
      if (cl != call) fail();
      if (w == 1) st_agent(me + OFF_CNT + b * LINE, call);
    }
  }
  asm volatile("" ::: "memory");
  STAMP(3);

  const auto rrs = __builtin_amdgcn_make_buffer_rsrc(p.ws[slot] + OFF_XC + par * XC_BYTES, 0, XC_BYTES, 0x00020000);
  const auto ors = __builtin_amdgcn_make_buffer_rsrc(p.out, 0, 4096 * NL * 4, 0x00020000);
  const u32 fi32 = (u32)(grp * 8 + w);
  const u32 vr = fi32 * FRAG + lane * 16;
  const float scale = *(const CST float*)p.sx * *(const CST float*)p.sw;
  auto put_partial = [&](const v4f* acc, int buf) {
#pragma unroll
    for (int c = 0; c < 4; ++c) *(LDS v4f*)(smem + s01 + buf * PBUF + r16 * PROW + c * 64 + gq * 16) = acc[c];
    u64 keep;
    asm volatile("s_mov_b64 %0, exec\n\ts_mov_b64 exec, 1\n\tds_add_u32 %1, %2\n\ts_mov_b64 exec, %0"
                 : "=&s"(keep) : "v"(lbase + L_CTL + 8 + buf * 8), "v"(1u) : "memory");
  };
  const u32 ov = ((u32)(grp * 16 + 2 * w + (lane >> 5)) * NL + panel * 64 + (lane & 31) * 2) * 4u | ((lane & 1) ? 0x80000000u : 0u);
  auto reduce = [&](int row0, int t) {
    const int buf = t & 1;
    LDS char* a = smem + L_STG + buf * PBUF + (2 * w + (lane >> 5)) * PROW + (lane & 31) * 8;
    v2f q[8];
    u32x2 ctl;
    while (true) {
      ctl = __builtin_bit_cast(u32x2, __hip_atomic_load((LDS u64*)(smem + L_CTL + 8 + buf * 8), __ATOMIC_RELAXED, __HIP_MEMORY_SCOPE_WORKGROUP));
      asm volatile("" ::: "memory");
#pragma unroll
      for (int i = 0; i < 8; ++i) q[i] = *(LDS v2f*)(a + i * L_STAGE);
      asm volatile("" : "+v"(q[0]), "+v"(q[1]), "+v"(q[2]), "+v"(q[3]), "+v"(q[4]), "+v"(q[5]), "+v"(q[6]), "+v"(q[7]));
      if (__builtin_amdgcn_readfirstlane(ctl.x) >= 8u * (u32)((t >> 1) + 1)) break;
      while (lds_ld(smem + L_CTL + 8 + buf * 8) < 8u * (u32)((t >> 1) + 1)) __builtin_amdgcn_s_sleep(1);
      asm volatile("" ::: "memory");
    }
    const v2f s = ((q[0] + q[1]) + (q[2] + q[3])) + ((q[4] + q[5]) + (q[6] + q[7]));
    const float sc_ = __builtin_amdgcn_readfirstlane(ctl.y) ? __builtin_nanf("") : scale;
    const v2f y = s * sc_;
    v2f d;
#pragma unroll
    for (int i = 0; i < 2; ++i) d[i] = __builtin_amdgcn_exp2f(__builtin_amdgcn_fmed3f(y[i], -60.f, 60.f) * -1.44269504f);
    d += 1.f;
    const v2f o = y * v2f{__builtin_amdgcn_rcpf(d[0]), __builtin_amdgcn_rcpf(d[1])};
    const v2i oi = __builtin_bit_cast(v2i, o);
    const int o0 = oi.x, o1 = oi.y;
    const u32x4 v = {(u32)o0, (u32)o1, (u32)__builtin_amdgcn_mov_dpp(o0, 0xb1, 0xf, 0xf, true), (u32)__builtin_amdgcn_mov_dpp(o1, 0xb1, 0xf, 0xf, true)};
    __builtin_amdgcn_raw_buffer_store_b128(v, ors, ov + (u32)row0 * (NL * 4), 0, 17);
    asm volatile("s_nop 1" :: "v"(v));
  };

  if (slot == 0) {
    const auto lrs = __builtin_amdgcn_make_buffer_rsrc(me, 0, WS_USED, 0x00020000);
    u32x4 nx[8];
    auto ldt = [&](int t) {
      const u32 so = t < 7 ? OFF_XC + par * XC_BYTES + XC_RAW + (u32)(t + 1) * (128 * FRAG) : OFF_XP + (par * NR + ((p.rank + t - 7) & 7)) * XP_BYTES;
#pragma unroll
      for (int i = 0; i < 8; ++i) nx[i] = __builtin_amdgcn_raw_buffer_load_b128(lrs, vr + (i & 1) * 1024, so + (i >> 1) * (32 * FRAG), 17);
    };
    auto row0 = [&](int t) { return t < 7 ? p.rank * 512 + (t + 1) * 64 : ((p.rank + t - 7) & 7) * 512; };
    ldt(0);
    for (int t = 0; t < 15; ++t) {
      u32x4 cu[8];
#pragma unroll
      for (int i = 0; i < 8; ++i) { cu[i] = nx[i]; asm volatile("" : "+v"(cu[i])); }
      if (t == 6) {
        const char* fl = me + OFF_PSH + gq * LINE + r16 * 4;
        if (poll_all(fl, fl + 4 * LINE, call, tmo, call) == ~0u && lane == 0) fail();
      }
      asm volatile("" ::: "memory");
      if (t < 14) ldt(t + 1);
      asm volatile("" ::: "memory");
      __builtin_amdgcn_sched_barrier(0);
      v4f acc[4] = {};
#pragma unroll
      for (int j = 0; j < 4; ++j) {
        const u32x4 a = cu[2 * j], c2 = cu[2 * j + 1];
        const v8i xv = {(int)a.x, (int)a.y, (int)a.z, (int)a.w, (int)c2.x, (int)c2.y, (int)c2.z, (int)c2.w};
#pragma unroll
        for (int c = 0; c < 4; ++c) acc[c] = __builtin_amdgcn_mfma_scale_f32_16x16x128_f8f6f4(Wf[j][c], xv, acc[c], 0, 0, 0, 0, 0, 0);
      }
      if (t > 0) reduce(row0(t - 1), t - 1);
      put_partial(acc, t & 1);
      if (t < 7) STAMP(5 + t);
    }
    reduce(row0(14), 14);
  } else {
    const u32 OOB = 0x80000000u;
    const u32 bs = (u32)lane * STEP + (u32)grp * BLK;
    const u32 tl = len && lane < 32 ? (bs + BLK_L + lst * 4) | ((len - 1u) << 24) : XC_BYTES;
    const u32 ta = lane >= 32 ? XC_BYTES : len ? bs + w * 1024 : (XC_RAW + (u32)lane * (32 * FRAG) + fi32 * FRAG) | 0x80000000u;
    const u32 v16 = lane * 16, v8 = lane * 8, v4 = lane * 4, vt = lane < TOUCH ? lane * 64 : OOB;
    u32x4 A, B;
    u32x2 C;
    u32 L, pf = 0;
    u32 vl, sc, sl2, sa, sb, sp;
    auto plan = [&](int step) {
      const u32 e = __builtin_amdgcn_readlane(tl, step + 1), a = __builtin_amdgcn_readlane(ta, step);
      const u32 ma = (u32)((int)a >> 31);
      const u32 a1 = __builtin_amdgcn_readlane(ta, step + 1);
      sp = a1 & 0xbfffffffu;
      sc = sp + BLK_C - w * 512;
      sl2 = e & 0xffffffu;
      vl = min(v4, (e >> 22) & 0xfcu);
      sa = a & 0x7fffffffu;
      sb = ma ? sa : XC_BYTES;
      return ma != 0u;
    };
    auto ld_cl = [&]() {
      C = __builtin_amdgcn_raw_buffer_load_b64(rrs, v8, sc, 0);
      L = __builtin_amdgcn_raw_buffer_load_b32(rrs, vl, sl2, 0);
    };
    auto ld_ab = [&]() {
      A = __builtin_amdgcn_raw_buffer_load_b128(rrs, v16, sa, 0);
      B = __builtin_amdgcn_raw_buffer_load_b128(rrs, v16 + 1024, sb, 0);
      if (TOUCH) {
        asm volatile("" :: "v"(pf));
        pf = __builtin_amdgcn_raw_buffer_load_b32(rrs, vt, sp, 0);
      }
    };
    plan(3);
    ld_cl();
    u32x2 c1 = C;
    u32 l1 = L;
    asm volatile("" ::: "memory");
    bool raw1 = plan(4);
    ld_cl();
    ld_ab();
    asm volatile("" ::: "memory");
    __builtin_amdgcn_sched_barrier(0);
    v8i ex = exponents(c1, l1, smem + L_LUT);
    for (int r = 1; r < 8; ++r) {
      v4f acc[4] = {};
#pragma unroll
      for (int j = 0; j < 4; ++j) {
        const int step = r * 4 + j;
        const bool raw = raw1;
        raw1 = plan(step + 1);
        asm volatile("" : "+v"(ex), "+v"(vl), "+s"(sc), "+s"(sl2), "+s"(sa), "+s"(sb), "+s"(sp));
        __builtin_amdgcn_sched_barrier(0);
        u32x4 a = A, b2 = B;
        c1 = C; l1 = L;
        asm volatile("" : "+v"(a), "+v"(b2), "+v"(c1), "+v"(l1));
        if (r == SR) STAMP(25 + j);
        ld_cl();
        ld_ab();
        asm volatile("" ::: "memory");
        __builtin_amdgcn_sched_barrier(0);
        if (raw) {
#pragma unroll
          for (int i = 0; i < 7; ++i) asm volatile("v_mov_b32 %0, %1" : "=v"(ex[i]) : "v"(i < 4 ? a[i] : b2[i - 4]));
          asm volatile("v_mov_b32 %0, %1\n\ts_nop 1" : "=v"(ex[7]) : "v"(b2[3]));
        } else {
          ex = merge(a, ex);
        }
#pragma unroll
        for (int c = 0; c < 4; ++c) acc[c] = __builtin_amdgcn_mfma_scale_f32_16x16x128_f8f6f4(Wf[j][c], ex, acc[c], 0, 0, 0, 0, 0, 0);
        if (j == 1 && r > 1) reduce(sh * 512 + (r - 1) * 64, r - 2);
        if (j == 3) put_partial(acc, (r - 1) & 1);
        if (step < 31 && !raw1) ex = exponents(c1, l1, smem + L_LUT);
      }
      STAMP(4 + r);
    }
    reduce(sh * 512 + 7 * 64, 6);
  }
  STAMP(12);
  STAMP(13);
  STAMP_FLUSH();
}

extern "C" void kernel_launch(int rank, int nranks, ncclComm_t comm,
                              hipStream_t stream,
                              void* const* d_in, const int* in_sizes, int n_in,
                              void* d_out, int out_size,
                              const int* in_dtypes, int out_dtype,
                              void* const* d_out_all,
                              void* const* d_ws_all, size_t ws_bytes) {
  if (nranks != NR || (unsigned)rank >= (unsigned)NR || n_in != 4 || ws_bytes < (size_t)WS_USED + (1u << 20) ||
      in_sizes[0] != 512 * KD || in_sizes[1] != KD * NL || out_size != 4096 * NL) {
    fprintf(stderr, "unexpected problem shape\n");
    return;
  }
  Params p;
  for (int i = 0; i < NR; ++i) p.ws[i] = (char*)d_ws_all[(rank + i) & 7];
  p.x = (const unsigned char*)d_in[0];
  p.w = (const unsigned char*)d_in[1];
  p.sx = (const float*)d_in[2];
  p.sw = (const float*)d_in[3];
  p.out = (float*)d_out;
  p.rank = rank;
  p.pad = 0;
  ag_gemm<<<256, 512, 0, stream>>>(p);
  HIP_CHECK(hipGetLastError());
}
```

```cpp
#include <hip/hip_runtime.h>
#include <rccl/rccl.h>
#include <cstdio>

#define HIP_CHECK(expr) do { hipError_t _e = (expr); if (_e != hipSuccess) { \
    fprintf(stderr, "HIP error %s at %s:%d\n", hipGetErrorString(_e), __FILE__, __LINE__); \
    return; } } while (0)

typedef unsigned u32;
typedef unsigned long long u64;
typedef u32 u32x4 __attribute__((ext_vector_type(4)));
typedef u32 u32x2 __attribute__((ext_vector_type(2)));
typedef int v2i __attribute__((ext_vector_type(2)));
typedef int v8i __attribute__((ext_vector_type(8)));
typedef float v4f __attribute__((ext_vector_type(4)));
typedef float v2f __attribute__((ext_vector_type(2)));
#define LDS __attribute__((address_space(3)))
#define GLB __attribute__((address_space(1)))
#define CST __attribute__((address_space(4)))

constexpr int NR = 8;
constexpr int KD = 4096, NL = 512;
constexpr u32 FRAG = 2048;
constexpr u32 LINE = 128;
constexpr u32 OFF_CNT = 0;
constexpr u32 OFF_TMO = OFF_CNT + 256 * LINE;
constexpr u32 OFF_ELECT = OFF_TMO + LINE;
constexpr u32 OFF_CLEAN = OFF_ELECT + 64 * LINE;
constexpr u32 OFF_CPY = OFF_CLEAN + 64 * LINE;
constexpr u32 CPY_GO = 112 * 8;
constexpr u32 OFF_PSH = OFF_CPY + 2 * NR * 1024;
constexpr u32 L2_DW = 63;
constexpr u32 BLK_C = 8 * 1024, BLK_L = 8 * 1536;
constexpr u32 BLK = BLK_L + 2048;
constexpr u32 STEP = 4 * BLK;
constexpr u32 OFF_XC = 1u << 20;
constexpr u32 XC_RAW = 2u << 20, XC_BYTES = XC_RAW + 1024 * FRAG;
constexpr u32 OFF_XP = OFF_XC + 2 * XC_BYTES;
constexpr u32 XP_BYTES = 128 * FRAG;
constexpr u32 WS_USED = OFF_XP + 2 * NR * XP_BYTES;
static_assert(32 * STEP <= XC_RAW && STEP % 128 == 0, "layout");
constexpr u32 L_LUT = 0;
constexpr u32 L_STG = 2048, L_STAGE = 16384;
constexpr u32 L_XTRA = L_STG + 8 * L_STAGE;
constexpr u32 L_CTL = L_XTRA + 8 * FRAG;
constexpr u32 L_BYTES = L_CTL + 64;
constexpr u32 PROW = 272;
constexpr u32 PBUF = 16 * PROW;
constexpr int PW = 3;
constexpr int GOS = 8;
constexpr int TOUCH = 2;
constexpr int HOLDW = 24;
constexpr int SELFD = 100;
constexpr int ENDRAW = 1;
constexpr int D1 = 10;
constexpr int DWIN = 16;
constexpr u64 DEADLINE_TICKS = 50000000ull;

struct Params {
  char* ws[NR];
  const unsigned char* x;
  const unsigned char* w;
  const float* sx;
  const float* sw;
  float* out;
  int rank;
  int pad;
};

template <int N> __device__ __forceinline__ void wait_vm() { asm volatile("s_waitcnt vmcnt(%0)" :: "n"(N) : "memory"); }
__device__ __forceinline__ void wait_lgkm() { asm volatile("s_waitcnt lgkmcnt(0)" ::: "memory"); }
__device__ __forceinline__ u32 ld_sys(const char* a) { return __hip_atomic_load((GLB u32*)a, __ATOMIC_RELAXED, __HIP_MEMORY_SCOPE_SYSTEM); }
__device__ __forceinline__ u32 ld_agent(const char* a) { return __hip_atomic_load((GLB u32*)a, __ATOMIC_RELAXED, __HIP_MEMORY_SCOPE_AGENT); }
__device__ __forceinline__ void st_agent(char* a, u32 v) { __hip_atomic_store((GLB u32*)a, v, __ATOMIC_RELAXED, __HIP_MEMORY_SCOPE_AGENT); }
__device__ __forceinline__ u32 lds_ld(LDS char* a) { return __hip_atomic_load((LDS u32*)a, __ATOMIC_RELAXED, __HIP_MEMORY_SCOPE_WORKGROUP); }

__device__ __forceinline__ void dma_w(u32 lds, u32 ve, u32 vo, u32x4 rs, u32 so) {
  u32 keep;
#define DMA_STEP(V) "s_add_u32 m0, m0, 0x400\n\ts_add_u32 %1, %1, 0x2000\n\tbuffer_load_dwordx4 " V ", %5, %1 offen lds\n\t"
  asm volatile("s_mov_b32 %0, m0\n\ts_mov_b32 m0, %2\n\ts_nop 0\n\tbuffer_load_dwordx4 %3, %5, %1 offen lds\n\t"
               DMA_STEP("%4") DMA_STEP("%3") DMA_STEP("%4") DMA_STEP("%3") DMA_STEP("%4") DMA_STEP("%3") DMA_STEP("%4")
               "s_mov_b32 m0, %0"
               : "=&s"(keep), "+s"(so) : "s"(lds), "v"(ve), "v"(vo), "s"(rs) : "memory", "scc");
#undef DMA_STEP
}
__device__ __forceinline__ void st16_sys(char* base, u32 off, u32x4 v) {
  asm volatile("global_store_dwordx4 %0, %1, %2 sc0 sc1\n\ts_nop 1" :: "v"(off), "v"(v), "s"(base) : "memory");
}
__device__ __forceinline__ void st8_sys(char* base, u32 off, u32x2 v) {
  asm volatile("global_store_dwordx2 %0, %1, %2 sc0 sc1\n\ts_nop 1" :: "v"(off), "v"(v), "s"(base) : "memory");
}
__device__ __forceinline__ void st4_sys(char* base, u32 off, u32 v) {
  asm volatile("global_store_dword %0, %1, %2 sc0 sc1\n\ts_nop 1" :: "v"(off), "v"(v), "s"(base) : "memory");
}
__device__ __forceinline__ u32 poll_all(const char* word, const char* word2, u32 target, char* tmo, u32 call) {
  const u64 t0 = __builtin_amdgcn_s_memrealtime();
  for (u32 it = 1;; ++it) {
    const u32 v = ld_sys(word), v2 = word2 ? ld_sys(word2) : v;
    if (__all(v >= target && v2 >= target)) return __builtin_amdgcn_readfirstlane(v);
    if ((it & 15u) == 0u && (ld_agent(tmo) == call || __builtin_amdgcn_s_memrealtime() - t0 > DEADLINE_TICKS)) { st_agent(tmo, call); return ~0u; }
    __builtin_amdgcn_s_sleep(1);
  }
}

__device__ __forceinline__ u64 poll_tag(const char* word, const char* word2, char* tmo, u32 call) {
  const u64 t0 = __builtin_amdgcn_s_memrealtime();
  for (u32 it = 1;; ++it) {
    const u64 v = __hip_atomic_load((GLB u64*)word, __ATOMIC_RELAXED, __HIP_MEMORY_SCOPE_SYSTEM);
    const u64 v2 = word2 ? __hip_atomic_load((GLB u64*)word2, __ATOMIC_RELAXED, __HIP_MEMORY_SCOPE_SYSTEM) : v;
    if (__all((u32)(v >> 48) == (call & 0xffffu) && (u32)(v2 >> 48) == (call & 0xffffu))) return v;
    if ((it & 15u) == 0u && (ld_agent(tmo) == call || __builtin_amdgcn_s_memrealtime() - t0 > DEADLINE_TICKS)) { st_agent(tmo, call); return (u64)(~call) << 48; }
    __builtin_amdgcn_s_sleep(1);
  }
}

constexpr u32 L2_CAP = (L2_DW * 32 - 8) / 3;

__device__ __forceinline__ u32 wave_scan(u32 v) {
  v += __builtin_amdgcn_update_dpp(0u, v, 0x111, 0xf, 0xf, false);
  v += __builtin_amdgcn_update_dpp(0u, v, 0x112, 0xf, 0xf, false);
  v += __builtin_amdgcn_update_dpp(0u, v, 0x114, 0xf, 0xf, false);
  v += __builtin_amdgcn_update_dpp(0u, v, 0x118, 0xf, 0xf, false);
  v += __builtin_amdgcn_update_dpp(0u, v, 0x142, 0xa, 0xf, false);
  v += __builtin_amdgcn_update_dpp(0u, v, 0x143, 0xc, 0xf, false);
  return v;
}
__device__ __forceinline__ void lut_entry(u32 cb, u32& dsel, u32& csel, u32& s3) {
  u32 n = 0; dsel = 0; csel = 0x0c0c0c0cu;
#pragma unroll
  for (int b = 0; b < 4; ++b) {
    const u32 c = (cb >> (2 * b)) & 3u;
    const bool e = c == 3u;
    dsel |= (e ? 4u + n : c) << (8 * b);
    csel = e ? (csel & ~(0xffu << (8 * n))) | ((u32)b << (8 * n)) : csel;
    n += e;
  }
  s3 = 3u * n;
}
struct Packed { u32x4 n; u32x2 c; u32 l; };

template <int B> __device__ __forceinline__ u32 byte_shl(u32 c, u32 sh) {
  u32 r;
  if (B == 0) asm("v_lshlrev_b32_sdwa %0, %1, %2 dst_sel:DWORD dst_unused:UNUSED_PAD src0_sel:DWORD src1_sel:BYTE_0" : "=v"(r) : "v"(sh), "v"(c));
  if (B == 1) asm("v_lshlrev_b32_sdwa %0, %1, %2 dst_sel:DWORD dst_unused:UNUSED_PAD src0_sel:DWORD src1_sel:BYTE_1" : "=v"(r) : "v"(sh), "v"(c));
  if (B == 2) asm("v_lshlrev_b32_sdwa %0, %1, %2 dst_sel:DWORD dst_unused:UNUSED_PAD src0_sel:DWORD src1_sel:BYTE_2" : "=v"(r) : "v"(sh), "v"(c));
  if (B == 3) asm("v_lshlrev_b32_sdwa %0, %1, %2 dst_sel:DWORD dst_unused:UNUSED_PAD src0_sel:DWORD src1_sel:BYTE_3" : "=v"(r) : "v"(sh), "v"(c));
  return r;
}
__device__ __forceinline__ u32 encode(const v8i& xi, Packed& o, LDS char* lutc, LDS char* buf, int lane) {
  u32 x[8];
#pragma unroll
  for (int i = 0; i < 8; ++i) x[i] = (u32)xi[i];
  u32 se = 0;
#pragma unroll
  for (int i = 0; i < 8; i += 2) se = __builtin_amdgcn_sad_u8(x[i] & 0x78787878u, 0u, se);
  const u32 tot = __builtin_amdgcn_readlane(wave_scan(se), 63);
  const u32 e0 = min((tot < 256u ? 0u : tot - 256u) >> 13, 10u);
  const u32 bias = (13u - e0) * 0x01010101u;
  u32 inv1 = 0, inv2 = 0, ca[8], kb[8];
#pragma unroll
  for (int i = 0; i < 8; ++i) {
    const u32 gb = ((x[i] >> 3) & 0x0f0f0f0fu) + bias;
    const u32 sel = gb & 0x07070707u;
    const u32 a = __builtin_amdgcn_perm(0x1008001cu, 0x1b1a1918u, sel);
    const u32 b = __builtin_amdgcn_perm(0x98989898u, 0x981f1e1du, sel);
    const u32 m = __builtin_amdgcn_perm(0u, 0u, __builtin_amdgcn_bitop3_b32(gb >> 4, 0x01010101u, 0x0c0c0c0cu, 0xea));
    const u32 code = __builtin_amdgcn_bitop3_b32(m, b, a, 0xca);
    inv1 = __builtin_amdgcn_bitop3_b32(inv1, gb, gb >> 1, 0xf9);
    inv2 |= code;
    ca[i] = __builtin_amdgcn_udot4(code & 0x18181818u, 0x40100401u, 0u, false);
    kb[i] = code & 0x07070707u;
  }
  u64 acc = 0; u32 n3 = 0;
#pragma unroll
  for (int i = 7; i >= 0; --i) {
    const u32x2 t = *(LDS u32x2*)(lutc + ca[i]);
    const u32 comp = __builtin_amdgcn_perm(kb[i], kb[i], t.x);
    acc = (acc << t.y) | __builtin_amdgcn_udot4(comp, 0x00400801u, byte_shl<3>(comp, 9u), false);
    n3 += t.y;
  }
  const u32 incl = wave_scan(n3);
  const u32 all3 = __builtin_amdgcn_readlane(incl, 63);
  const bool fit = !__any(((inv1 & 0x08080808u) | (inv2 & 0x80808080u)) != 0u || n3 > 63u) && all3 <= 3u * L2_CAP;
  o.c = u32x2{(ca[0] >> 3) | (ca[1] << 5) | (ca[2] << 13) | (ca[3] << 21), (ca[4] >> 3) | (ca[5] << 5) | (ca[6] << 13) | (ca[7] << 21)};
#pragma unroll
  for (int j = 0; j < 4; ++j) {
    const u32 od = x[2 * j + 1] & 0x87878787u;
    o.n[j] = (x[2 * j] & 0x87878787u) | __builtin_amdgcn_alignbit(od, od, 28);
  }
  o.l = 0;
  if (!fit) return 0u;
  const u32 off = incl - n3 + 8u, sh = off & 31u;
  const u64 lo = acc << sh;
  const u32 hi = ((u32)(acc >> 32) >> 1) >> (31u - sh);
  LDS u32* d = (LDS u32*)(buf + ((off >> 5) << 2));
  __hip_atomic_fetch_or(d, (u32)lo | (lane == 0 ? e0 : 0u), __ATOMIC_RELAXED, __HIP_MEMORY_SCOPE_WAVEFRONT);
  __hip_atomic_fetch_or(d + 1, (u32)(lo >> 32), __ATOMIC_RELAXED, __HIP_MEMORY_SCOPE_WAVEFRONT);
  __hip_atomic_fetch_or(d + 2, hi, __ATOMIC_RELAXED, __HIP_MEMORY_SCOPE_WAVEFRONT);
  o.l = __hip_atomic_load((LDS u32*)(buf + lane * 4), __ATOMIC_RELAXED, __HIP_MEMORY_SCOPE_WAVEFRONT);
  return (all3 + 8u + 31u) >> 5;
}

__device__ __forceinline__ v8i exponents(u32x2 c, u32 l, LDS char* lutd) {
  u32x2 t[8];
  t[0] = *(LDS u32x2*)(lutd + byte_shl<0>(c.x, 3u)); t[1] = *(LDS u32x2*)(lutd + byte_shl<1>(c.x, 3u));
  t[2] = *(LDS u32x2*)(lutd + byte_shl<2>(c.x, 3u)); t[3] = *(LDS u32x2*)(lutd + byte_shl<3>(c.x, 3u));
  t[4] = *(LDS u32x2*)(lutd + byte_shl<0>(c.y, 3u)); t[5] = *(LDS u32x2*)(lutd + byte_shl<1>(c.y, 3u));
  t[6] = *(LDS u32x2*)(lutd + byte_shl<2>(c.y, 3u)); t[7] = *(LDS u32x2*)(lutd + byte_shl<3>(c.y, 3u));
  __builtin_amdgcn_sched_barrier(0);
  const u32 n = __builtin_popcount(__builtin_amdgcn_bitop3_b32(c.x, c.x >> 1, 0x55555555u, 0x80)) +
                __builtin_popcount(__builtin_amdgcn_bitop3_b32(c.y, c.y >> 1, 0x55555555u, 0x80));
  u32 incl = wave_scan(n);
  asm("" : "+v"(incl));
  const u32 off = __umul24(incl - n, 3u) + 8u;
  const int a = (int)(off >> 3);
  const u32 w0 = __builtin_amdgcn_ds_bpermute(a, l), w1 = __builtin_amdgcn_ds_bpermute(a + 4, l), w2 = __builtin_amdgcn_ds_bpermute(a + 8, l);
  u64 win = ((u64)__builtin_amdgcn_alignbit(w2, w1, off) << 32) | __builtin_amdgcn_alignbit(w1, w0, off);
  const u32 e8 = (__builtin_amdgcn_readfirstlane(l) & 15u) * 0x08080808u;
  const u32 td = 0x00100800u + e8, tl = (0x70686058u + e8) & 0x7f7f7f7fu, th = (0xa8a09878u + e8) & 0x7f7f7f7fu;
  v8i o;
#pragma unroll
  for (int i = 0; i < 8; ++i) {
    const u32 a0 = (u32)win;
    const u32 y = __builtin_amdgcn_perm(a0 >> 6, a0, 0x0c040c00u);
    const u32 z = __builtin_amdgcn_bitop3_b32(y, y << 5, 0x07070707u, 0xa8);
    o[i] = (int)__builtin_amdgcn_perm(__builtin_amdgcn_perm(th, tl, z), td, t[i].x);
    if (i < 7) win >>= t[i].y;
  }
  return o;
}
__device__ __forceinline__ v8i merge(u32x4 n, v8i ex) {
#pragma unroll
  for (int i = 0; i < 8; ++i) {
    const u32 nb = (i & 1) ? __builtin_amdgcn_alignbit(n[i >> 1], n[i >> 1], 4) : n[i >> 1];
    ex[i] = (int)__builtin_amdgcn_bitop3_b32(nb, 0x87878787u, (u32)ex[i], 0xea);
  }
  return ex;
}

template <int WIN> __device__ __forceinline__ void drive(const char* xc, u32 lds, int lane, int n) {
  const u64 xa = (u64)xc;
  const u32x4 xrs = {(u32)xa, (u32)(xa >> 32) & 0xffffu, XC_RAW, 0x00020000u};
  u32 keep;
  for (int i = 0; i < n * 4; ++i) {
    const u32 bi = (u32)(i >> 2), li = (i & 3) * 27 + lane;
    const u32 vo = lane < 27 && li < 107 ? 4 * STEP + bi * BLK + li * 128 : 0x80000000u;
    asm volatile("s_mov_b32 %0, m0\n\ts_mov_b32 m0, %1\n\ts_waitcnt vmcnt(%4)\n\tbuffer_load_dword %2, %3, 0 offen lds\n\ts_mov_b32 m0, %0"
                 : "=&s"(keep) : "s"(lds), "v"(vo), "s"(xrs), "n"(WIN - 1) : "memory");
  }
}

__global__ void __launch_bounds__(512, 2) ag_gemm(Params p) {
  __shared__ __attribute__((aligned(16))) char smem_[L_BYTES];
  LDS char* const smem = (LDS char*)smem_;
  const int lane = threadIdx.x & 63;
  const int w = __builtin_amdgcn_readfirstlane(threadIdx.x >> 6);
  const int b = blockIdx.x, slot = b & 7, q = b >> 3, grp = q >> 3, panel = q & 7;
  const int gq = lane >> 4, r16 = lane & 15;
  const unsigned char* px;
  char *me, *dst;
  const int pc = slot * 2 + (q >> 4);
  const int di = w == 7 ? 0 : (w + pc) % 7 + 1;
  asm volatile("s_load_dwordx2 %0, %3, 0x40\n\ts_load_dwordx2 %1, %3, 0x0\n\ts_load_dwordx2 %2, %3, %4"
               : "=&s"(px), "=&s"(me), "=&s"(dst) : "s"(__builtin_amdgcn_kernarg_segment_ptr()), "s"(di * 8));
  const bool job = panel < 4, pushcu = (q & 15) == 0;
  const int jt = grp * 4 + panel;
  const bool coord = jt == 1;
  const int Fp = w * 16 + ((pc + 2 * w) & 15), Fc = 128 + (slot * 14 + jt - 1 - (jt > 8)) * 8 + w;
  const int F = Fc + ((Fp - Fc) & -(int)pushcu);
  u32 xo = (u32)((F >> 7) * 64 + ((F >> 3) & 3) * 16 + r16) * KD + ((((F >> 5) & 3) * 8 + (F & 7)) * 128 + gq * 16);
  asm volatile("s_waitcnt lgkmcnt(0)" : "+v"(xo), "+s"(px), "+s"(me), "+s"(dst));
  const char* const cntp = me + OFF_CNT + b * LINE;
  u32 call;
  u32x4 x0, x1;
  if (job)
    asm volatile("global_load_dwordx4 %0, %3, %5\n\tglobal_load_dwordx4 %1, %3, %5 offset:64\n\ts_load_dword %2, %4, 0x0"
                 : "=&v"(x0), "=&v"(x1), "=&s"(call) : "v"(xo), "s"(cntp), "s"(px) : "memory");
  const u64 wa = (u64)p.w;
  const u32x4 wrs = {(u32)wa, (u32)(wa >> 32) & 0xffffu, (u32)(KD * NL), 0x00020000u};
  const u32 wve = (u32)(lane >> 2) * NL + panel * 64 + (((lane & 3) ^ (gq & 1)) << 4), wvo = wve ^ 32u;
  const u32 lbase = (u32)(size_t)smem;
  const u32 s01 = L_STG + w * L_STAGE;
  v8i Wf[4][4];
  const int q8 = r16 >> 1, f2 = ((q8 >> 2) & 1) | ((gq & 1) << 1);
  const u32 tra = (u32)(gq * 16 + q8) * 64 + (r16 & 1) * 8;
  auto read_stage = [&](int j, u32 off) {
#pragma unroll
    for (int c = 0; c < 4; ++c) {
      LDS char* a = smem + off + tra + ((c ^ f2) << 4);
      v2i t0 = __builtin_amdgcn_ds_read_tr8_b64_v2i32((LDS v2i*)(a));
      v2i t1 = __builtin_amdgcn_ds_read_tr8_b64_v2i32((LDS v2i*)(a + 512));
      v2i t2 = __builtin_amdgcn_ds_read_tr8_b64_v2i32((LDS v2i*)(a + 4096));
      v2i t3 = __builtin_amdgcn_ds_read_tr8_b64_v2i32((LDS v2i*)(a + 4608));
      Wf[j][c] = v8i{t0.x, t0.y, t1.x, t1.y, t2.x, t2.y, t3.x, t3.y};
    }
  };
  const int sh = (p.rank + slot) & 7;
  char* const tmo = me + OFF_TMO;
  u32 xcc;
  asm volatile("s_getreg_b32 %0, hwreg(HW_REG_XCC_ID)" : "=s"(xcc));
  const u32 eo = ((xcc & 7u) * 8 + slot) * LINE;
  auto lead = [&]() {
    u32 old = 0;
    if (lane == 0) old = __hip_atomic_fetch_max((GLB u32*)(me + OFF_ELECT + eo), call, __ATOMIC_RELAXED, __HIP_MEMORY_SCOPE_AGENT);
    if (__builtin_amdgcn_readfirstlane(old) >= call) return 0u;
    const char* fl = me + OFF_CPY + ((call & 1u) * NR + sh) * 1024;
    const u32 cl = __builtin_amdgcn_readfirstlane((u32)(poll_tag(fl + lane * 8, fl + 512 + lane * 8, tmo, call) >> 48)) == (call & 0xffffu) ? call : ~0u;
    __builtin_amdgcn_fence(__ATOMIC_ACQUIRE, "");
    wait_vm<0>();
    if (lane == 0) st_agent(me + OFF_CLEAN + eo, cl);
    return cl;
  };
  auto tables = [&]() {
    if (w < 4) { u32 d, c, n; lut_entry(threadIdx.x, d, c, n); *(LDS u32x2*)(smem + L_LUT + threadIdx.x * 8) = u32x2{d, n}; }
    if (threadIdx.x < 8) *(LDS u32*)(smem + L_CTL + threadIdx.x * 4) = 0u;
  };
  if (job) {
    if (!pushcu) {
#pragma unroll
      for (int i = 0; i < 4; ++i) {
        u32 d, c, n;
        lut_entry(lane + 64 * i, d, c, n);
        *(LDS u32x2*)(smem + s01 + (lane + 64 * i) * 8) = u32x2{c, n};
      }
      *(LDS u32*)(smem + s01 + 2048 + lane * 4) = 0u;
    }
    asm volatile("s_waitcnt vmcnt(0) lgkmcnt(0)" : "+v"(x0), "+v"(x1), "+s"(call) :: "memory");
    call += 1u;
    LDS char* sl = smem + L_XTRA + w * FRAG;
    if (__builtin_expect(pushcu, 1)) {
      const u32 off = OFF_XP + ((call & 1u) * NR + p.rank) * XP_BYTES + lane * 16;
      st16_sys(dst, off + F * FRAG, x0); st16_sys(dst, off + F * FRAG + 1024, x1);
      *(LDS u32x4*)(sl + lane * 16) = x0;
      *(LDS u32x4*)(sl + 1024 + lane * 16) = x1;
      wait_lgkm();
      __builtin_amdgcn_s_barrier();
      asm volatile("" ::: "memory");
      if (w == 7) __builtin_amdgcn_s_sleep(SELFD);
#pragma unroll
      for (int i = 1; i < 8; ++i) {
        const int fi = (i + w) & 7, Fi = fi * 16 + ((pc + 2 * fi) & 15);
        const u32x4 a = *(LDS u32x4*)(smem + L_XTRA + fi * FRAG + lane * 16), c = *(LDS u32x4*)(smem + L_XTRA + fi * FRAG + 1024 + lane * 16);
#pragma unroll
        for (int h = 0; h < 2; ++h) {
          const int k = 2 * i + h;
          if (k >= PW) { if (k > GOS && k < GOS + PW) wait_vm<PW>(); else wait_vm<PW - 1>(); }
          if (k == GOS && lane == 0) st8_sys(dst, OFF_CPY + ((call & 1u) * NR + p.rank) * 1024 + CPY_GO + pc * 8, u32x2{0u, call << 16});
          st16_sys(dst, off + Fi * FRAG + h * 1024, h ? c : a);
        }
      }
      wait_vm<0>();
      if (lane == 0) st4_sys(dst, OFF_PSH + p.rank * LINE + pc * 4, call);
      tables();
      wait_lgkm();
      __builtin_amdgcn_s_barrier();
    } else {
      const u32 base = OFF_XC + (call & 1u) * XC_BYTES;
      st16_sys(me, base + XC_RAW + F * FRAG + lane * 16, x0); st16_sys(me, base + XC_RAW + F * FRAG + 1024 + lane * 16, x1);
      Packed pk;
      const u32 len = F >= 1024 - 32 * ENDRAW ? 0u : encode(v8i{(int)x0.x, (int)x0.y, (int)x0.z, (int)x0.w, (int)x1.x, (int)x1.y, (int)x1.z, (int)x1.w}, pk, smem + s01, smem + s01 + 2048, lane);
      const u32 blk = base + (u32)(F >> 3) * BLK;
      if (lane == 0) *(LDS u32*)(smem + L_CTL + 32 + w * 4) = len;
      if (len) {
        *(LDS u32x2*)(sl + lane * 8) = pk.c;
        st16_sys(me, blk + w * 1024 + lane * 16, pk.n);
        const u32x4 c = *(LDS u32x4*)(sl + lane * 16);
        if (lane < 32) st16_sys(me, blk + BLK_C + w * 512 + lane * 16, c);
      }
      wait_lgkm();
      __builtin_amdgcn_s_barrier();
      asm volatile("" ::: "memory");
      const u32 ln = lane < 8 ? *(LDS u32*)(smem + L_CTL + 32 + lane * 4) : 0u, st = wave_scan(ln) - ln;
      u32 lo = blk + BLK_L + ((u32)__builtin_amdgcn_readlane(st, w) + lane) * 4;
      asm volatile("" : "+v"(lo));
      if (lane < (int)len) st4_sys(me, lo, pk.l);
      u64 fo = (u64)(call & 0xffffu) << 48;
#pragma unroll
      for (int i = 0; i < 8; ++i) fo |= (u64)(u32)__builtin_amdgcn_readlane(ln, i) << (6 * i);
      tables();
      char* wsl = p.ws[0];
#pragma unroll
      for (int i = 1; i < NR; ++i) wsl = (lane & 7) == i ? p.ws[i] : wsl;
      wsl += OFF_CPY + ((call & 1u) * NR + p.rank) * 1024 + ((F - 128) >> 3) * 8;
      asm volatile("s_waitcnt vmcnt(0) lgkmcnt(0)" : "+v"(wsl) :: "memory");
      __builtin_amdgcn_s_barrier();
      if (w == 0) {
        if (lane < NR) asm volatile("global_store_dwordx2 %0, %1, off sc0 sc1" :: "v"(wsl), "v"(fo) : "memory");
        if (coord && lead() != 0u && slot != 0) drive<DWIN>(p.ws[slot] + base, lbase + L_XTRA, lane, D1);
      }
      if (coord) __builtin_amdgcn_s_barrier();
    }
  }
  u32 cl0 = 0;
  u64 fv;
  const char* lenp;
  {
    wait_lgkm();
    if (!job) {
      __builtin_amdgcn_s_sleep(HOLDW);
      u32 keep;
      const u32 pv = (u32)(((slot * 16 + grp * 4 + panel - 4) * 8 + w) * 2048 + lane * 16);
      asm volatile("s_mov_b32 %0, m0\n\ts_mov_b32 m0, %1\n\ts_nop 0\n\tbuffer_load_dwordx4 %2, %3, 0 offen lds\n\t"
                   "buffer_load_dwordx4 %2, %3, 0 offen offset:1024 lds\n\ts_mov_b32 m0, %0\n\ts_waitcnt vmcnt(0)"
                   : "=&s"(keep) : "s"(lbase + L_XTRA + w * FRAG), "v"(pv), "s"(wrs) : "memory");
    }
    dma_w(lbase + s01, wve, wvo, wrs, (u32)w * 65536u);
    dma_w(lbase + s01 + 8192, wve, wvo, wrs, (u32)(w + 8) * 65536u);
    if (!job) {
      tables();
      asm volatile("s_load_dword %0, %1, 0x0\n\ts_waitcnt lgkmcnt(0)" : "=&s"(call) : "s"(cntp) : "memory");
      call += 1u;
      __builtin_amdgcn_s_barrier();
    }
    wait_vm<8>();
    read_stage(0, s01);
    wait_lgkm();
    dma_w(lbase + s01, wve, wvo, wrs, (u32)(w + 16) * 65536u);
    wait_vm<8>();
    read_stage(1, s01 + 8192);
    wait_lgkm();
    dma_w(lbase + s01 + 8192, wve, wvo, wrs, (u32)(w + 24) * 65536u);
    lenp = me + OFF_CPY + ((call & 1u) * NR + sh) * 1024 + ((min(max(lane, 4), 31) - 4) * 4 + grp) * 8;
    fv = __hip_atomic_load((GLB u64*)lenp, __ATOMIC_RELAXED, __HIP_MEMORY_SCOPE_SYSTEM);
    cl0 = ld_agent(me + OFF_CLEAN + eo);
    wait_vm<8>();
    read_stage(2, s01);
    wait_vm<0>();
    read_stage(3, s01 + 8192);
  }
  call = __builtin_amdgcn_readfirstlane(call);
  const u32 par = call & 1u;
  auto fail = [&]() { *(LDS u32*)(smem + L_CTL + 12) = 1u; *(LDS u32*)(smem + L_CTL + 20) = 1u; };
  u32 len = 0, lst = 0;
  if (slot != 0) {
    if (!__all((u32)(fv >> 48) == (call & 0xffffu))) {
      fv = poll_tag(lenp, nullptr, tmo, call);
      if (!__all((u32)(fv >> 48) == (call & 0xffffu)) && lane == 0) fail();
    }
#pragma unroll
    for (int i = 0; i < 7; ++i) lst += i < w ? (u32)(fv >> (6 * i)) & 63u : 0u;
    len = (u32)(fv >> (6 * w)) & 63u;
    asm volatile("" : "+v"(len), "+v"(lst));
  }
  {
    u32 cl = __builtin_amdgcn_readfirstlane(cl0);
    for (int i = 0; i < 16 && cl < call; ++i) cl = __builtin_amdgcn_readfirstlane(ld_agent(me + OFF_CLEAN + eo));
    if (cl < call && w == 1) cl = lead();
    if (cl < call) cl = poll_all(me + OFF_CLEAN + eo, nullptr, call, tmo, call);
    if (lane == 0) {
      if (cl != call) fail();
      if (w == 1) st_agent(me + OFF_CNT + b * LINE, call);
    }
  }
  asm volatile("" ::: "memory");

  const auto rrs = __builtin_amdgcn_make_buffer_rsrc(p.ws[slot] + OFF_XC + par * XC_BYTES, 0, XC_BYTES, 0x00020000);
  const auto ors = __builtin_amdgcn_make_buffer_rsrc(p.out, 0, 4096 * NL * 4, 0x00020000);
  const u32 fi32 = (u32)(grp * 8 + w);
  const u32 vr = fi32 * FRAG + lane * 16;
  const float scale = *(const CST float*)p.sx * *(const CST float*)p.sw;
  auto put_partial = [&](const v4f* acc, int buf) {
#pragma unroll
    for (int c = 0; c < 4; ++c) *(LDS v4f*)(smem + s01 + buf * PBUF + r16 * PROW + c * 64 + gq * 16) = acc[c];
    u64 keep;
    asm volatile("s_mov_b64 %0, exec\n\ts_mov_b64 exec, 1\n\tds_add_u32 %1, %2\n\ts_mov_b64 exec, %0"
                 : "=&s"(keep) : "v"(lbase + L_CTL + 8 + buf * 8), "v"(1u) : "memory");
  };
  const u32 ov = ((u32)(grp * 16 + 2 * w + (lane >> 5)) * NL + panel * 64 + (lane & 31) * 2) * 4u | ((lane & 1) ? 0x80000000u : 0u);
  auto reduce = [&](int row0, int t) {
    const int buf = t & 1;
    LDS char* a = smem + L_STG + buf * PBUF + (2 * w + (lane >> 5)) * PROW + (lane & 31) * 8;
    v2f q[8];
    u32x2 ctl;
    while (true) {
      ctl = __builtin_bit_cast(u32x2, __hip_atomic_load((LDS u64*)(smem + L_CTL + 8 + buf * 8), __ATOMIC_RELAXED, __HIP_MEMORY_SCOPE_WORKGROUP));
      asm volatile("" ::: "memory");
#pragma unroll
      for (int i = 0; i < 8; ++i) q[i] = *(LDS v2f*)(a + i * L_STAGE);
      asm volatile("" : "+v"(q[0]), "+v"(q[1]), "+v"(q[2]), "+v"(q[3]), "+v"(q[4]), "+v"(q[5]), "+v"(q[6]), "+v"(q[7]));
      if (__builtin_amdgcn_readfirstlane(ctl.x) >= 8u * (u32)((t >> 1) + 1)) break;
      while (lds_ld(smem + L_CTL + 8 + buf * 8) < 8u * (u32)((t >> 1) + 1)) __builtin_amdgcn_s_sleep(1);
      asm volatile("" ::: "memory");
    }
    const v2f s = ((q[0] + q[1]) + (q[2] + q[3])) + ((q[4] + q[5]) + (q[6] + q[7]));
    const float sc_ = __builtin_amdgcn_readfirstlane(ctl.y) ? __builtin_nanf("") : scale;
    const v2f y = s * sc_;
    v2f d;
#pragma unroll
    for (int i = 0; i < 2; ++i) d[i] = __builtin_amdgcn_exp2f(__builtin_amdgcn_fmed3f(y[i], -60.f, 60.f) * -1.44269504f);
    d += 1.f;
    const v2f o = y * v2f{__builtin_amdgcn_rcpf(d[0]), __builtin_amdgcn_rcpf(d[1])};
    const v2i oi = __builtin_bit_cast(v2i, o);
    const int o0 = oi.x, o1 = oi.y;
    const u32x4 v = {(u32)o0, (u32)o1, (u32)__builtin_amdgcn_mov_dpp(o0, 0xb1, 0xf, 0xf, true), (u32)__builtin_amdgcn_mov_dpp(o1, 0xb1, 0xf, 0xf, true)};
    __builtin_amdgcn_raw_buffer_store_b128(v, ors, ov + (u32)row0 * (NL * 4), 0, 17);
    asm volatile("s_nop 1" :: "v"(v));
  };

  if (slot == 0) {
    const auto lrs = __builtin_amdgcn_make_buffer_rsrc(me, 0, WS_USED, 0x00020000);
    u32x4 nx[8];
    auto ldt = [&](int t) {
      const u32 so = t < 7 ? OFF_XC + par * XC_BYTES + XC_RAW + (u32)(t + 1) * (128 * FRAG) : OFF_XP + (par * NR + ((p.rank + t - 7) & 7)) * XP_BYTES;
#pragma unroll
      for (int i = 0; i < 8; ++i) nx[i] = __builtin_amdgcn_raw_buffer_load_b128(lrs, vr + (i & 1) * 1024, so + (i >> 1) * (32 * FRAG), 17);
    };
    auto row0 = [&](int t) { return t < 7 ? p.rank * 512 + (t + 1) * 64 : ((p.rank + t - 7) & 7) * 512; };
    ldt(0);
    for (int t = 0; t < 15; ++t) {
      u32x4 cu[8];
#pragma unroll
      for (int i = 0; i < 8; ++i) { cu[i] = nx[i]; asm volatile("" : "+v"(cu[i])); }
      if (t == 6) {
        const char* fl = me + OFF_PSH + gq * LINE + r16 * 4;
        if (poll_all(fl, fl + 4 * LINE, call, tmo, call) == ~0u && lane == 0) fail();
      }
      asm volatile("" ::: "memory");
      if (t < 14) ldt(t + 1);
      asm volatile("" ::: "memory");
      __builtin_amdgcn_sched_barrier(0);
      v4f acc[4] = {};
#pragma unroll
      for (int j = 0; j < 4; ++j) {
        const u32x4 a = cu[2 * j], c2 = cu[2 * j + 1];
        const v8i xv = {(int)a.x, (int)a.y, (int)a.z, (int)a.w, (int)c2.x, (int)c2.y, (int)c2.z, (int)c2.w};
#pragma unroll
        for (int c = 0; c < 4; ++c) acc[c] = __builtin_amdgcn_mfma_scale_f32_16x16x128_f8f6f4(Wf[j][c], xv, acc[c], 0, 0, 0, 0, 0, 0);
      }
      if (t > 0) reduce(row0(t - 1), t - 1);
      put_partial(acc, t & 1);
    }
    reduce(row0(14), 14);
  } else {
    const u32 OOB = 0x80000000u;
    const u32 bs = (u32)lane * STEP + (u32)grp * BLK;
    const u32 tl = len && lane < 32 ? (bs + BLK_L + lst * 4) | ((len - 1u) << 24) : XC_BYTES;
    const u32 ta = lane >= 32 ? XC_BYTES : len ? bs + w * 1024 : (XC_RAW + (u32)lane * (32 * FRAG) + fi32 * FRAG) | 0x80000000u;
    const u32 v16 = lane * 16, v8 = lane * 8, v4 = lane * 4, vt = lane < TOUCH ? lane * 64 : OOB;
    u32x4 A, B;
    u32x2 C;
    u32 L, pf = 0;
    u32 vl, sc, sl2, sa, sb, sp;
    auto plan = [&](int step) {
      const u32 e = __builtin_amdgcn_readlane(tl, step + 1), a = __builtin_amdgcn_readlane(ta, step);
      const u32 ma = (u32)((int)a >> 31);
      const u32 a1 = __builtin_amdgcn_readlane(ta, step + 1);
      sp = a1 & 0xbfffffffu;
      sc = sp + BLK_C - w * 512;
      sl2 = e & 0xffffffu;
      vl = min(v4, (e >> 22) & 0xfcu);
      sa = a & 0x7fffffffu;
      sb = ma ? sa : XC_BYTES;
      return ma != 0u;
    };
    auto ld_cl = [&]() {
      C = __builtin_amdgcn_raw_buffer_load_b64(rrs, v8, sc, 0);
      L = __builtin_amdgcn_raw_buffer_load_b32(rrs, vl, sl2, 0);
    };
    auto ld_ab = [&]() {
      A = __builtin_amdgcn_raw_buffer_load_b128(rrs, v16, sa, 0);
      B = __builtin_amdgcn_raw_buffer_load_b128(rrs, v16 + 1024, sb, 0);
      asm volatile("" :: "v"(pf));
      pf = __builtin_amdgcn_raw_buffer_load_b32(rrs, vt, sp, 0);
    };
    plan(3);
    ld_cl();
    u32x2 c1 = C;
    u32 l1 = L;
    asm volatile("" ::: "memory");
    bool raw1 = plan(4);
    ld_cl();
    ld_ab();
    asm volatile("" ::: "memory");
    __builtin_amdgcn_sched_barrier(0);
    v8i ex = exponents(c1, l1, smem + L_LUT);
    for (int r = 1; r < 8; ++r) {
      v4f acc[4] = {};
#pragma unroll
      for (int j = 0; j < 4; ++j) {
        const int step = r * 4 + j;
        const bool raw = raw1;
        raw1 = plan(step + 1);
        asm volatile("" : "+v"(ex), "+v"(vl), "+s"(sc), "+s"(sl2), "+s"(sa), "+s"(sb), "+s"(sp));
        __builtin_amdgcn_sched_barrier(0);
        u32x4 a = A, b2 = B;
        c1 = C; l1 = L;
        asm volatile("" : "+v"(a), "+v"(b2), "+v"(c1), "+v"(l1));
        ld_cl();
        ld_ab();
        asm volatile("" ::: "memory");
        __builtin_amdgcn_sched_barrier(0);
        if (raw) {
#pragma unroll
          for (int i = 0; i < 7; ++i) asm volatile("v_mov_b32 %0, %1" : "=v"(ex[i]) : "v"(i < 4 ? a[i] : b2[i - 4]));
          asm volatile("v_mov_b32 %0, %1\n\ts_nop 1" : "=v"(ex[7]) : "v"(b2[3]));
        } else {
          ex = merge(a, ex);
        }
#pragma unroll
        for (int c = 0; c < 4; ++c) acc[c] = __builtin_amdgcn_mfma_scale_f32_16x16x128_f8f6f4(Wf[j][c], ex, acc[c], 0, 0, 0, 0, 0, 0);
        if (j == 1 && r > 1) reduce(sh * 512 + (r - 1) * 64, r - 2);
        if (j == 3) put_partial(acc, (r - 1) & 1);
        if (step < 31 && !raw1) ex = exponents(c1, l1, smem + L_LUT);
      }
    }
    reduce(sh * 512 + 7 * 64, 6);
  }
}

extern "C" void kernel_launch(int rank, int nranks, ncclComm_t comm,
                              hipStream_t stream,
                              void* const* d_in, const int* in_sizes, int n_in,
                              void* d_out, int out_size,
                              const int* in_dtypes, int out_dtype,
                              void* const* d_out_all,
                              void* const* d_ws_all, size_t ws_bytes) {
  if (nranks != NR || (unsigned)rank >= (unsigned)NR || n_in != 4 || ws_bytes < (size_t)WS_USED + (1u << 20) ||
      in_sizes[0] != 512 * KD || in_sizes[1] != KD * NL || out_size != 4096 * NL) {
    fprintf(stderr, "unexpected problem shape\n");
    return;
  }
  Params p;
  for (int i = 0; i < NR; ++i) p.ws[i] = (char*)d_ws_all[(rank + i) & 7];
  p.x = (const unsigned char*)d_in[0];
  p.w = (const unsigned char*)d_in[1];
  p.sx = (const float*)d_in[2];
  p.sw = (const float*)d_in[3];
  p.out = (float*)d_out;
  p.rank = rank;
  p.pad = 0;
  ag_gemm<<<256, 512, 0, stream>>>(p);
  HIP_CHECK(hipGetLastError());
}
```
